```python
import numpy as np

kernel_name = "gpu_ag_gemm_m4096_k4096_n4096_f32_none"


ABSMAX_RTOL = 0.001

COMPUTE_DTYPE = "float32"
EPILOGUE = "none"
IN_QUANT = None
OUT_QUANT = None
IN_DTYPES = {"x": "float32", "w_mat": "float32"}
WORLD_SIZE = 8

def setup_inputs():

    x = np.random.randn(4096, 4096).astype(np.float32)
    w_mat = (np.random.randn(4096, 4096) / np.sqrt(4096)).astype(np.float32)
    return {"x": x, "w_mat": w_mat}

def _epilogue(y):
    return y

def reference(x, w_mat):
    y = (x @ w_mat).astype(np.float32)
    return _epilogue(y)
```

```cpp
#include <hip/hip_runtime.h>
#include <rccl/rccl.h>
#include <cstdio>

#define HIP_CHECK(expr) do { hipError_t _e = (expr); if (_e != hipSuccess) { \
    fprintf(stderr, "HIP error %s at %s:%d\n", hipGetErrorString(_e), __FILE__, __LINE__); \
    return; } } while (0)

namespace {

typedef __attribute__((address_space(1))) unsigned gu32;
typedef unsigned u32x4 __attribute__((ext_vector_type(4)));
typedef unsigned u32x2 __attribute__((ext_vector_type(2)));
typedef float f32x4 __attribute__((ext_vector_type(4)));
typedef _Float16 h16x2 __attribute__((ext_vector_type(2)));
typedef _Float16 h16x8 __attribute__((ext_vector_type(8)));

#define P_R1 23
#define P_E 2
#define P_NS 107
#define P_NW 32
constexpr int R1 = P_R1;
constexpr int R2 = 32 - R1;
constexpr int E = P_E;
constexpr int NREP = 7;
constexpr int NW = P_NW;
constexpr int NOWN = 32 + 3 * R1 + 4 * R2;
constexpr int NFOR = 4 * R1 + 3 * R2;
constexpr int NS = P_NS;
constexpr int NI = (R1 - E) * 32;
constexpr int X20 = 256;
constexpr int O0 = X20 + (R2 - E) * 32;
constexpr int NC = O0 + 2 * NS;
constexpr int NR = NI > NC ? NI : NC;
constexpr int MAXL = 24;
static_assert(NS <= NFOR && NFOR <= 128 && (NOWN + NREP - 1) / NREP <= MAXL && NW <= 32 && NW % 8 == 0, "");

constexpr unsigned MiB = 1u << 20;
constexpr unsigned CNT_OFF = 0;
constexpr unsigned TMO_OFF = 16 << 10;
constexpr unsigned XF_OFF = 64 << 10;
constexpr unsigned WF_OFF = 128 << 10;
constexpr unsigned OF_OFF = 160 << 10;
constexpr unsigned DF_OFF = 192 << 10;
constexpr unsigned PAR_STRIDE = 40 * MiB;
constexpr unsigned XS_OFF = 2 * MiB;
constexpr unsigned WS_OFF = XS_OFF + 32 * MiB;
constexpr unsigned OS_OFF = 82 * MiB;
constexpr unsigned DUMMY_OFF = 86 * MiB;
constexpr unsigned STAMP_OFF = 88 * MiB;
constexpr unsigned WS_USED = 90 * MiB;
constexpr unsigned long long DEADLINE = 30000000ull;

struct Tables { unsigned strip[128]; unsigned char order[5][NREP * MAXL]; };
constexpr int arrival(int d, int rb) {
  if (d == 0) return -100;
  if (d < 4) return rb < E ? -1 : (rb - E + 1) * 32;
  return rb - R1 < E ? -1 : X20 + (rb - R1 - E + 1) * 32;
}
constexpr bool wanted(int type, int d, int rb) {
  if (d == 0) return type == 0 || rb < R1;
  if (d < 4) return rb < R1;
  return rb >= R1 && (type == 0 || d != 3 + type);
}
constexpr Tables make_tables() {
  Tables t{};
  for (int type = 0; type < 5; ++type) {
    int n = 0;
    for (int rb = 0; rb < 32; ++rb)
      for (int d = 0; d < 8; ++d)
        if (wanted(type, d, rb)) {
          int e = d * 32 + rb, a = arrival(d, rb), i = n++;
          while (i > 0 && arrival(t.order[type][i - 1] >> 5, t.order[type][i - 1] & 31) > a) {
            t.order[type][i] = t.order[type][i - 1];
            --i;
          }
          t.order[type][i] = (unsigned char)e;
        }
  }
  for (int k = 0; k < 128; ++k)
    for (int q = 0; q < 4; ++q) t.strip[k] |= (unsigned)t.order[1 + q][k] << (8 * q);
  return t;
}
constexpr bool margins_ok() {
  Tables t = make_tables();
  for (int type = 1; type < 5; ++type)
    for (int k = 0; k < NS; ++k) {
      int a = arrival(t.order[type][k] >> 5, t.order[type][k] & 31);
      if (a < X20) a = X20;
      if (a + 2 * NW > O0 + 2 * k) return false;
    }
  return true;
}
static_assert(margins_ok(), "forwarded strips must not wait for later rounds");
__constant__ const Tables TB = make_tables();

struct Args {
  const float* x;
  const float* w;
  char* ws[8];
  char* out[5];
  int rank;
  int pad;
};

#define WAITV(n) asm volatile("s_waitcnt vmcnt(%0)" ::"i"(n) : "memory")
#define CBAR() asm volatile("" ::: "memory")
constexpr int SYS = 17;

__device__ __forceinline__ unsigned pk(float a, float b) {
  h16x2 h = {(_Float16)a, (_Float16)b};
  return __builtin_bit_cast(unsigned, h);
}
__device__ __forceinline__ u32x4 pk8(f32x4 a, f32x4 b) {
  return u32x4{pk(a[0], a[1]), pk(a[2], a[3]), pk(b[0], b[1]), pk(b[2], b[3])};
}
__device__ __forceinline__ void st16(char* base, unsigned voff, u32x4 v) {
  asm volatile("global_store_dwordx4 %0, %1, %2 sc0 sc1\n\ts_nop 1" ::"v"(voff), "v"(v), "s"(base) : "memory");
}
__device__ __forceinline__ void put_flag(char* p, unsigned v) {
  __hip_atomic_store((gu32*)p, v, __ATOMIC_RELAXED, __HIP_MEMORY_SCOPE_SYSTEM);
}
__device__ __forceinline__ bool all_ge(u32x4 f, unsigned call) {
  return f[0] >= call && f[1] >= call && f[2] >= call && f[3] >= call;
}
__device__ __forceinline__ unsigned long long now() { return __builtin_amdgcn_s_memrealtime(); }

template <int SLEEP>
__device__ __forceinline__ bool spin4(__amdgpu_buffer_rsrc_t rs, char* ws, unsigned off, bool need, unsigned call,
                                      unsigned long long t0) {
  gu32* tmo = (gu32*)(ws + TMO_OFF);
  for (unsigned it = 0;; ++it) {
    bool ok = true;
    CBAR();
    if (need) ok = all_ge(__builtin_amdgcn_raw_buffer_load_b128(rs, off, 0, SYS), call);
    if (__all(ok)) { CBAR(); return true; }
    if ((it & 15) == 15) {
      if (__hip_atomic_load(tmo, __ATOMIC_RELAXED, __HIP_MEMORY_SCOPE_AGENT) == call || now() - t0 > DEADLINE) {
        __hip_atomic_store(tmo, call, __ATOMIC_RELAXED, __HIP_MEMORY_SCOPE_AGENT);
        return false;
      }
    }
    __builtin_amdgcn_s_sleep(SLEEP);
  }
}

#define STAMP(i) do { } while (0)

struct Round {
  unsigned srcI, srcC[4];
  unsigned dstC[4];
  unsigned stepC;
  int kindI, kindC;
  unsigned flagI, flagC, flagQ;
  unsigned rdyI, rdyC, rdyQ;
};
__device__ __forceinline__ Round make_round(int r, unsigned par, int pw, int rank) {
  Round R;
  const unsigned dummy = DUMMY_OFF + pw * 16384;
  if (r < NI) {
    unsigned rb = E + (r >> 5), s = r & 31;
    R.kindI = 1; R.srcI = XS_OFF + par + rb * 131072 + s * 4096;
    R.flagI = XF_OFF + (rb * 32 + s) * 4; R.rdyI = R.flagI & ~15u;
  } else {
    R.kindI = 0; R.srcI = dummy; R.flagI = 0; R.rdyI = XF_OFF;
  }
  R.stepC = 1024;
  if (r < X20) {
    R.kindC = 1;
#pragma unroll
    for (int q = 0; q < 4; ++q) { R.srcC[q] = WS_OFF + par + q * MiB + r * 4096; R.dstC[q] = R.srcC[q] + 4 * MiB; }
    R.flagC = WF_OFF + (1024 + r) * 4; R.flagQ = 1024; R.rdyC = (WF_OFF + r * 4) & ~15u; R.rdyQ = 1024;
  } else if (r < O0) {
    unsigned rb = R1 + E + ((r - X20) >> 5), s = (r - X20) & 31;
    R.kindC = 1;
#pragma unroll
    for (int q = 0; q < 4; ++q) { R.srcC[q] = XS_OFF + par + rb * 131072 + s * 4096; R.dstC[q] = R.srcC[q] + (4 + q) * 4 * MiB; }
    R.flagC = XF_OFF + ((128 + rb) * 32 + s) * 4; R.flagQ = 4096; R.rdyC = (XF_OFF + (rb * 32 + s) * 4) & ~15u; R.rdyQ = 0;
  } else if (r < NC) {
    unsigned k = (r - O0) >> 1, h = (r - O0) & 1;
    R.kindC = 2; R.stepC = 4096;
    const unsigned e4 = TB.strip[k];
#pragma unroll
    for (int q = 0; q < 4; ++q) {
      unsigned e = (e4 >> (8 * q)) & 255;
      unsigned row = (rank ^ (e >> 5)) * 512 + (e & 31) * 16 + h * 8;
      R.srcC[q] = OS_OFF + (q * 128 + k) * 8192 + h * 4096;
      R.dstC[q] = row * 2048 + q * 512;
    }
    R.flagC = 0; R.flagQ = 0; R.rdyC = OF_OFF + k * 16; R.rdyQ = 2048;
  } else {
    R.kindC = 0;
#pragma unroll
    for (int q = 0; q < 4; ++q) { R.srcC[q] = dummy; R.dstC[q] = dummy; }
    R.flagC = 0; R.flagQ = 0; R.rdyC = XF_OFF; R.rdyQ = 0;
  }
  return R;
}

template <int F>
__device__ __forceinline__ void xfer3(u32x4& v, unsigned dst, char* b1, char* b2, char* b3, unsigned src, char* ws) {
  asm volatile(
      "global_store_dwordx4 %1, %0, %2 offset:%7 sc0 sc1\n\t"
      "global_store_dwordx4 %1, %0, %3 offset:%7 sc0 sc1\n\t"
      "global_store_dwordx4 %1, %0, %4 offset:%7 sc0 sc1\n\t"
      "s_nop 1\n\t"
      "global_load_dwordx4 %0, %5, %6 offset:%7 sc0 sc1"
      : "+v"(v) : "v"(dst), "s"(b1), "s"(b2), "s"(b3), "v"(src), "s"(ws), "i"(F * 1024) : "memory");
}
template <int F>
__device__ __forceinline__ void xfer1(u32x4& v, unsigned dst, char* b, unsigned src, char* ws) {
  asm volatile(
      "global_store_dwordx4 %1, %0, %2 sc0 sc1\n\t"
      "s_nop 1\n\t"
      "global_load_dwordx4 %0, %3, %4 offset:%5 sc0 sc1"
      : "+v"(v) : "v"(dst), "s"(b), "v"(src), "s"(ws), "i"(F * 1024) : "memory");
}
template <int F>
__device__ __forceinline__ void ld16(u32x4& v, unsigned src, char* ws) {
  asm volatile("global_load_dwordx4 %0, %1, %2 offset:%3 sc0 sc1" : "+v"(v) : "v"(src), "s"(ws), "i"(F * 1024) : "memory");
}

__device__ __forceinline__ void pusher(const Args& a, int pw, int lane, int wave, unsigned call, unsigned long long t0) {
  char* const ws = a.ws[0];
  const unsigned par = (call & 1) * PAR_STRIDE;
  const __amdgpu_buffer_rsrc_t rs = __builtin_amdgcn_make_buffer_rsrc(ws, 0, WS_USED, 0x00020000);
  char* const pl = lane == 1 ? a.ws[1] : lane == 2 ? a.ws[2] : lane == 3 ? a.ws[3] : lane == 4 ? a.ws[4]
                 : lane == 5 ? a.ws[5] : lane == 6 ? a.ws[6] : a.ws[7];
  const unsigned l16 = lane * 16, lout = (lane >> 5) * 2048 + (lane & 31) * 16;
  auto rdy_off = [&](const Round& R) { return lane == 4 ? R.rdyI : R.rdyC + (lane & 3) * R.rdyQ; };
  auto publish = [&](unsigned fI, unsigned fC, unsigned fQ) {
    unsigned off = lane < 4 ? fI + lane * 4096 : fC + (lane - 4) * fQ;
    bool on = lane >= 1 && lane < 8 && (lane < 4 ? fI : fC) != 0;
    if (on) put_flag(pl + off, call);
  };
  u32x4 vI[4], vC[4][4], rd = {0, 0, 0, 0};
#pragma unroll
  for (int f = 0; f < 4; ++f) { vI[f] = rd; vC[f][0] = rd; vC[f][1] = rd; vC[f][2] = rd; vC[f][3] = rd; }

  Round cur = make_round(pw, par, pw, a.rank), nxt = make_round(pw + NW, par, pw, a.rank);
  if (!spin4<1>(rs, ws, rdy_off(cur), lane < 5, call, t0)) return;
  STAMP(1);
#define LDSTEP(f) ld16<f>(vI[f], cur.srcI + l16, ws); ld16<f>(vC[f][0], cur.srcC[0] + l16, ws); ld16<f>(vC[f][1], cur.srcC[1] + l16, ws); \
                  ld16<f>(vC[f][2], cur.srcC[2] + l16, ws); ld16<f>(vC[f][3], cur.srcC[3] + l16, ws);
  ld16<0>(rd, rdy_off(nxt), ws);
  LDSTEP(0) LDSTEP(1) LDSTEP(2) LDSTEP(3)
#undef LDSTEP
  WAITV(0);
  unsigned pI = 0, pC = 0, pQ = 0;
  for (int r = pw; r < NR; r += NW) {
    WAITV(36);
    asm volatile("" : "+v"(rd));
    if (!__all(lane >= 5 || all_ge(rd, call))) {
      WAITV(0);
      publish(pI, pC, pQ);
      pI = pC = 0;
      if (!spin4<1>(rs, ws, rdy_off(nxt), lane < 5, call, t0)) return;
    }
    const Round nn = make_round(r + 2 * NW, par, pw, a.rank);
    ld16<0>(rd, rdy_off(nn), ws);
    char* const b1 = cur.kindI ? a.ws[1] + 4 * MiB : ws;
    char* const b2 = cur.kindI ? a.ws[2] + 8 * MiB : ws;
    char* const b3 = cur.kindI ? a.ws[3] + 12 * MiB : ws;
    char* const c0 = cur.kindC == 1 ? a.ws[4] : cur.kindC == 2 ? a.out[1] : ws;
    char* const c1 = cur.kindC == 1 ? a.ws[5] : cur.kindC == 2 ? a.out[2] : ws;
    char* const c2 = cur.kindC == 1 ? a.ws[6] : cur.kindC == 2 ? a.out[3] : ws;
    char* const c3 = cur.kindC == 1 ? a.ws[7] : cur.kindC == 2 ? a.out[4] : ws;
    const unsigned lc = cur.kindC == 2 ? lout : l16;
    const unsigned dI = cur.srcI + l16, sI = nxt.srcI + l16;
    const unsigned s0 = nxt.srcC[0] + l16, s1 = nxt.srcC[1] + l16, s2 = nxt.srcC[2] + l16, s3 = nxt.srcC[3] + l16;
#define STEP(f) xfer1<f>(vC[f][0], cur.dstC[0] + f * cur.stepC + lc, c0, s0, ws); \
                xfer1<f>(vC[f][1], cur.dstC[1] + f * cur.stepC + lc, c1, s1, ws); \
                xfer3<f>(vI[f], dI, b1, b2, b3, sI, ws);                          \
                xfer1<f>(vC[f][2], cur.dstC[2] + f * cur.stepC + lc, c2, s2, ws); \
                xfer1<f>(vC[f][3], cur.dstC[3] + f * cur.stepC + lc, c3, s3, ws);
    asm volatile("s_nop 4" :: "s"(b1), "s"(b2), "s"(b3), "s"(c0), "s"(c1), "s"(c2), "s"(c3), "s"(ws));
    STEP(0)
    WAITV(36);
    STEP(1)
    WAITV(36);
    STEP(2)
    WAITV(36);
    publish(pI, pC, pQ);
    STEP(3)
#undef STEP
    pI = cur.flagI; pC = cur.flagC; pQ = cur.flagQ;
    cur = nxt; nxt = nn;
  }
  WAITV(0);
  publish(pI, pC, pQ);
  if (lane >= 4 && lane < 8) put_flag(pl + DF_OFF + (lane - 4) * 256 + pw * 4, call);
  STAMP(5);
}

__device__ __forceinline__ void load_x(f32x4 (&s)[8], const float* x, int rb, int slot, int lane) {
  const f32x4* p = (const f32x4*)(x + (size_t)(rb * 16 + (lane & 15)) * 4096 + slot * 128 + (lane >> 4) * 8);
#pragma unroll
  for (int f = 0; f < 4; ++f) { s[2 * f] = p[8 * f]; s[2 * f + 1] = p[8 * f + 1]; }
}

constexpr unsigned OOB = 0xF0000000u;
template <int IMM>
__device__ __forceinline__ void ldbuf(u32x4& v, __amdgpu_buffer_rsrc_t rs, unsigned voff, unsigned soff) {
  asm volatile("buffer_load_dwordx4 %0, %1, %2, %3 offen offset:%4 sc0 sc1" : "+v"(v) : "v"(voff), "s"(rs), "s"(soff), "i"(IMM) : "memory");
}
__device__ __forceinline__ void load_block(u32x4 (&A)[16], __amdgpu_buffer_rsrc_t rs, unsigned voff, unsigned soff) {
  asm volatile("s_nop 4");
#define L4(j) ldbuf<0>(A[j], rs, voff, soff + j * 1024); ldbuf<1024>(A[j + 1], rs, voff, soff + j * 1024); \
              ldbuf<2048>(A[j + 2], rs, voff, soff + j * 1024); ldbuf<3072>(A[j + 3], rs, voff, soff + j * 1024);
  L4(0) L4(4) L4(8) L4(12)
#undef L4
}

__global__ void __launch_bounds__(512, 2) ag_gemm(const Args a) {
  extern __shared__ __attribute__((aligned(16))) char smem[];
  const int lane = threadIdx.x & 63;
  const int wave = __builtin_amdgcn_readfirstlane(threadIdx.x >> 6);
  const int xcd = blockIdx.x & 7, slot = blockIdx.x >> 3;
  char* const ws = a.ws[0];
  const unsigned long long t0 = now();
  gu32* const cnt = (gu32*)(ws + CNT_OFF) + blockIdx.x * 8 + wave;

  if (slot >= 28) {
    const int pw = (slot - 28) * 8 + xcd;
    if (wave != 0 || pw >= NW) return;
    const unsigned call = __builtin_amdgcn_readfirstlane(__hip_atomic_load(cnt, __ATOMIC_RELAXED, __HIP_MEMORY_SCOPE_AGENT)) + 1;
    __hip_atomic_store(cnt, call, __ATOMIC_RELAXED, __HIP_MEMORY_SCOPE_AGENT);
    STAMP(0);
    pusher(a, pw, lane, wave, call, t0);
    return;
  }

  const int id = xcd * 28 + slot, rep = id >> 5, cg = id & 31;
  const int cb = cg >> 2, g = cg & 3;
  const int type = cb < 4 ? 0 : cb - 3;
  const bool early = cb >= 4 && g == 3;
  const unsigned l16 = lane * 16;

  unsigned call;
  {
    f32x4 s[8], s2[8];
    int kind = 0, t;
    if (early) {
      t = wave * 28 + rep * 4 + (cb - 4);
      if (t < E * 32) { kind = 1; load_x(s, a.x, t >> 5, t & 31, lane); load_x(s2, a.x, R1 + (t >> 5), t & 31, lane); }
    } else {
      t = wave * 196 + rep * 28 + (cg < 16 ? cg : cg - (cb - 4));
      if (t < 1024) {
        kind = 2;
        const f32x4* p = (const f32x4*)(a.w + (size_t)((t & 63) * 64 + (lane >> 3) * 8) * 512 + (t >> 6) * 32 + (lane & 7) * 4);
#pragma unroll
        for (int j = 0; j < 8; ++j) s[j] = p[j * 128];
      } else if (t < 1024 + (32 - 2 * E) * 16) {
        kind = 3; t -= 1024;
        int bi = t >> 4, rb = bi < R1 - E ? bi + E : bi + 2 * E;
        t = rb * 32 + (t & 15) * 2;
        load_x(s, a.x, rb, t & 31, lane); load_x(s2, a.x, rb, (t & 31) + 1, lane);
      }
    }
    call = __builtin_amdgcn_readfirstlane(__hip_atomic_load(cnt, __ATOMIC_RELAXED, __HIP_MEMORY_SCOPE_AGENT)) + 1;
    __hip_atomic_store(cnt, call, __ATOMIC_RELAXED, __HIP_MEMORY_SCOPE_AGENT);
    STAMP(0);
    const unsigned par = (call & 1) * PAR_STRIDE;
    if (kind == 1) {
      const unsigned o1 = XS_OFF + par + t * 4096 + l16, o2 = o1 + R1 * 131072;
#pragma unroll
      for (int f = 0; f < 4; ++f) {
        const u32x4 h1 = pk8(s[2 * f], s[2 * f + 1]), h2 = pk8(s2[2 * f], s2[2 * f + 1]);
        st16(a.ws[4] + 16 * MiB, o2 + f * 1024, h2); st16(a.ws[1] + 4 * MiB, o1 + f * 1024, h1);
        st16(a.ws[5] + 20 * MiB, o2 + f * 1024, h2); st16(a.ws[2] + 8 * MiB, o1 + f * 1024, h1);
        st16(a.ws[6] + 24 * MiB, o2 + f * 1024, h2); st16(a.ws[3] + 12 * MiB, o1 + f * 1024, h1);
        st16(a.ws[7] + 28 * MiB, o2 + f * 1024, h2);
        st16(ws, o1 + f * 1024, h1); st16(ws, o2 + f * 1024, h2);
      }
      WAITV(0);
      char* const pl = lane == 1 ? a.ws[1] : lane == 2 ? a.ws[2] : lane == 3 ? a.ws[3] : lane == 4 ? a.ws[4]
                     : lane == 5 ? a.ws[5] : lane == 6 ? a.ws[6] : lane == 7 ? a.ws[7] : ws;
      if (lane < 9) put_flag(pl + XF_OFF + (lane & 7) * 4096 + (lane >= 4 ? R1 * 128 : 0) + t * 4, call);
    } else if (kind == 2) {
      const unsigned o = WS_OFF + par + t * 4096 + l16;
#pragma unroll
      for (int c = 0; c < 4; ++c)
        st16(ws, o + c * 1024, u32x4{pk(s[0][c], s[1][c]), pk(s[2][c], s[3][c]), pk(s[4][c], s[5][c]), pk(s[6][c], s[7][c])});
      WAITV(0);
      if (lane == 0) put_flag(ws + WF_OFF + t * 4, call);
    } else if (kind == 3) {
      const unsigned o = XS_OFF + par + t * 4096 + l16;
#pragma unroll
      for (int f = 0; f < 4; ++f) st16(ws, o + f * 1024, pk8(s[2 * f], s[2 * f + 1]));
#pragma unroll
      for (int f = 0; f < 4; ++f) st16(ws, o + 4096 + f * 1024, pk8(s2[2 * f], s2[2 * f + 1]));
      WAITV(0);
      if (lane < 2) put_flag(ws + XF_OFF + (t + lane) * 4, call);
    }
  }
  STAMP(1);
  const unsigned par = (call & 1) * PAR_STRIDE;
  const __amdgpu_buffer_rsrc_t rs = __builtin_amdgcn_make_buffer_rsrc(ws, 0, WS_USED, 0x00020000);

  bool good = spin4<8>(rs, ws, WF_OFF + (cg * 64 + wave * 8 + (lane & 1) * 4) * 4, lane < 2, call, t0);
  STAMP(2);
  h16x8 B[16][2];
  {
    const unsigned vo = (lane & 3) * 1024 + ((lane >> 4) * 8 + ((lane & 15) >> 2)) * 16;
    const unsigned so = WS_OFF + par + (cg * 64 + wave * 8) * 4096;
#pragma unroll
    for (int s = 0; s < 16; ++s)
#pragma unroll
      for (int t = 0; t < 2; ++t)
        B[s][t] = __builtin_bit_cast(h16x8, __builtin_amdgcn_raw_buffer_load_b128(rs, vo + (s & 1) * 512 + t * 64, so + (s >> 1) * 4096, SYS));
  }

  const int total = type == 0 ? NOWN : NFOR;
  const int n = (total - rep + NREP - 1) / NREP;
  const unsigned ev = TB.order[type][(lane < n ? lane : 0) * NREP + rep];
  const unsigned foff = XF_OFF + ev * 128 + wave * 16;
  const unsigned abase = XS_OFF + par + wave * 16384;
  char* const outp = type == 0 ? a.out[0] : type == 1 ? a.out[1] : type == 2 ? a.out[2] : type == 3 ? a.out[3] : a.out[4];
  const __amdgpu_buffer_rsrc_t ro = __builtin_amdgcn_make_buffer_rsrc(outp, 0, 8 * MiB, 0x00020000);
  const unsigned wslot = ((lane & 15) * 8) * 16, wrot = (lane >> 4) + (lane & 15);
  unsigned long long mask = 0;
  unsigned pend = 0;
  u32x4 A[16], pf = {0, 0, 0, 0};
#pragma unroll
  for (int j = 0; j < 16; ++j) A[j] = pf;
  good = spin4<1>(rs, ws, foff, lane == 0, call, t0) && good;
  load_block(A, rs, l16, abase + __builtin_amdgcn_readlane(ev, 0) * 131072);
  __builtin_amdgcn_s_waitcnt(0x0F70);
  for (int i = 0; i < n; ++i) {
    STAMP(8 + i);
    const unsigned e = __builtin_amdgcn_readlane(ev, i);
    const bool more = i + 1 < n;
    const unsigned nbase = abase + __builtin_amdgcn_readlane(ev, more ? i + 1 : i) * 131072;
    bool have = more && ((mask >> (i + 1)) & 1);
    const unsigned nb = have ? nbase : OOB;
    asm volatile("buffer_load_dwordx4 %0, %1, %2, 0 offen sc0 sc1\n\ts_nop 3" : "+v"(pf) : "v"(foff), "s"(rs) : "memory");
    f32x4 acc0 = {0, 0, 0, 0}, acc1 = {0, 0, 0, 0};
#define MSTEP(j) asm volatile("s_waitcnt vmcnt(16)" : "+v"(A[j]) :: "memory");                                   \
    acc0 = __builtin_amdgcn_mfma_f32_16x16x32_f16(B[j][0], __builtin_bit_cast(h16x8, A[j]), acc0, 0, 0, 0);      \
    acc1 = __builtin_amdgcn_mfma_f32_16x16x32_f16(B[j][1], __builtin_bit_cast(h16x8, A[j]), acc1, 0, 0, 0);      \
    __builtin_amdgcn_sched_barrier(0);                                                                           \
    ldbuf<(j & 3) * 1024>(A[j], rs, l16, nb + (j >> 2) * 4096);                                                  \
    __builtin_amdgcn_sched_barrier(0);
    MSTEP(0) MSTEP(1) MSTEP(2) MSTEP(3) MSTEP(4) MSTEP(5) MSTEP(6) MSTEP(7)
    MSTEP(8) MSTEP(9) MSTEP(10) MSTEP(11) MSTEP(12) MSTEP(13) MSTEP(14) MSTEP(15)
#undef MSTEP
    asm volatile("s_waitcnt vmcnt(16)" : "+v"(pf) :: "memory");
    mask = __ballot(all_ge(pf, call));
    if (!have && more && ((mask >> (i + 1)) & 1)) { have = true; load_block(A, rs, l16, nbase); }
    if (pend) { if (lane == 0) put_flag(ws + pend, call); pend = 0; }
    if (!good) { acc0 = __builtin_nanf(""); acc1 = acc0; }
    char* const pb = smem + (i & 1) * 16384;
    *(f32x4*)(pb + wave * 2048 + wslot + (wrot & 7) * 16) = acc0;
    *(f32x4*)(pb + wave * 2048 + wslot + ((wrot + 4) & 7) * 16) = acc1;
    asm volatile("s_waitcnt lgkmcnt(0)\n\ts_barrier" ::: "memory");
    if (wave == (i & 7)) {
      const int k = i * NREP + rep;
      const bool staged = type != 0 && k < NS;
      const unsigned row = (a.rank ^ (e >> 5)) * 512 + (e & 31) * 16;
#pragma unroll
      for (int h = 0; h < 2; ++h) {
        const unsigned m = h * 8 + (lane >> 3), c = lane & 7;
        const char* q = pb + (m * 8 + ((c + m) & 7)) * 16;
        f32x4 v = *(const f32x4*)q;
#pragma unroll
        for (int w = 1; w < 8; ++w) v += *(const f32x4*)(q + w * 2048);
        const u32x4 u = __builtin_bit_cast(u32x4, v);
        if (staged) __builtin_amdgcn_raw_buffer_store_b128(u, rs, m * 512 + c * 16, OS_OFF + ((type - 1) * 128 + k) * 8192 + g * 128, SYS);
        else __builtin_amdgcn_raw_buffer_store_b128(u, ro, m * 2048 + c * 16, row * 2048 + (cg & 15) * 128, SYS);
      }
      CBAR();
      if (staged) pend = OF_OFF + (((type - 1) * 128 + k) * 4 + g) * 4;
    }
    if (!have && more) {
      if (pend) { WAITV(0); if (lane == 0) put_flag(ws + pend, call); pend = 0; }
      good = spin4<1>(rs, ws, foff, lane == i + 1, call, t0) && good;
      load_block(A, rs, l16, nbase);
    }
  }
  WAITV(0);
  if (pend && lane == 0) put_flag(ws + pend, call);
  STAMP(3);
  if (type != 0) {
    asm volatile("s_barrier" ::: "memory");
    char* const pw = type == 1 ? a.ws[4] : type == 2 ? a.ws[5] : type == 3 ? a.ws[6] : a.ws[7];
    if (threadIdx.x == 0) put_flag(pw + DF_OFF + (type - 1) * 256 + (32 + rep * 4 + g) * 4, call);
  } else if (id == 0) {
    asm volatile("s_barrier" ::: "memory");
    if (wave == 0) {
      const int w4 = (lane & 15) * 4;
      const bool need = w4 < NW || (w4 >= 32 && w4 < 60);
      if (!spin4<1>(rs, ws, DF_OFF + lane * 16, need, call, t0)) {
        const unsigned e = __builtin_amdgcn_readlane(ev, 0);
        if (lane == 0) put_flag(a.out[0] + (size_t)((a.rank ^ (e >> 5)) * 512 + (e & 31) * 16) * 2048, 0x7fc00000u);
      }
      STAMP(4);
    }
  }
}

}

extern "C" void kernel_launch(int rank, int nranks, ncclComm_t comm, hipStream_t stream,
                              void* const* d_in, const int* in_sizes, int n_in,
                              void* d_out, int out_size, const int* in_dtypes, int out_dtype,
                              void* const* d_out_all, void* const* d_ws_all, size_t ws_bytes) {
  if (nranks != 8 || (unsigned)rank >= 8u || n_in < 2 || ws_bytes < WS_USED || out_size != 4096 * 512 ||
      in_sizes[0] != 512 * 4096 || in_sizes[1] != 4096 * 512) {
    fprintf(stderr, "unexpected problem shape\n");
    return;
  }
  constexpr int LDS_BYTES = 96 * 1024;
  HIP_CHECK(hipFuncSetAttribute((const void*)ag_gemm, hipFuncAttributeMaxDynamicSharedMemorySize, LDS_BYTES));
  Args a;
  a.x = (const float*)d_in[0];
  a.w = (const float*)d_in[1];
  for (int d = 0; d < 8; ++d) a.ws[d] = (char*)d_ws_all[rank ^ d];
  a.out[0] = (char*)d_out;
  for (int q = 0; q < 4; ++q) a.out[1 + q] = (char*)d_out_all[rank ^ (4 + q)];
  a.rank = rank;
  a.pad = 0;
  ag_gemm<<<256, 512, LDS_BYTES, stream>>>(a);
  HIP_CHECK(hipGetLastError());
}
```

```cpp
#include <hip/hip_runtime.h>
#include <rccl/rccl.h>
#include <cstdio>

#define HIP_CHECK(expr) do { hipError_t _e = (expr); if (_e != hipSuccess) { \
    fprintf(stderr, "HIP error %s at %s:%d\n", hipGetErrorString(_e), __FILE__, __LINE__); \
    return; } } while (0)

namespace {

typedef __attribute__((address_space(1))) unsigned gu32;
typedef unsigned u32x4 __attribute__((ext_vector_type(4)));
typedef unsigned u32x2 __attribute__((ext_vector_type(2)));
typedef float f32x4 __attribute__((ext_vector_type(4)));
typedef _Float16 h16x2 __attribute__((ext_vector_type(2)));
typedef _Float16 h16x8 __attribute__((ext_vector_type(8)));

#define SL1 1
#define SL2 8
#define EMASK 255
#define XDELAY 64
#define LOOK 3
#define SOFS(x) (x)
#define OAUX SYS
#define P_R1 23
#define P_XA 96
#define P_NS 104
#define P_NW 24
constexpr int R1 = P_R1;
constexpr int R2 = 32 - R1;
constexpr int XA = P_XA;
#define P_RO 5
constexpr int REPO = P_RO;
constexpr int REPF = 14 - REPO;
constexpr int NW = P_NW;
constexpr int NOWN = 32 + 3 * R1 + 4 * R2;
constexpr int NFOR = 4 * R1 + 3 * R2;
constexpr int NS = P_NS;
constexpr int NI = R1 * 32;
constexpr int X20 = XA + 256;
constexpr int O0 = 256 + R2 * 32;
constexpr int NC = O0 + 2 * NS;
constexpr int NR = NI > NC ? NI : NC;
static_assert(NS <= NFOR && NFOR <= 128 && NOWN <= 160 && (NOWN + REPO - 1) / REPO <= 60 && NW <= 32 && NW % 8 == 0 && REPF <= 12, "");

constexpr unsigned MiB = 1u << 20;
constexpr unsigned CNT_OFF = 0;
constexpr unsigned TMO_OFF = 16 << 10;
constexpr unsigned TICK_OFF = 20 << 10;
constexpr unsigned XF_OFF = 256 << 10;
constexpr unsigned WF_OFF = 512 << 10;
constexpr unsigned OF_OFF = 576 << 10;
constexpr unsigned DF_OFF = 640 << 10;
__host__ __device__ constexpr unsigned fl(unsigned s) { return (s >> 2) * 128 + (s & 3) * 4; }
constexpr unsigned PAR_STRIDE = 40 * MiB;
constexpr unsigned XS_OFF = 2 * MiB;
constexpr unsigned WS_OFF = XS_OFF + 32 * MiB;
constexpr unsigned OS_OFF = 82 * MiB;
constexpr unsigned DUMMY_OFF = 86 * MiB;
constexpr unsigned STAMP_OFF = 88 * MiB;
constexpr unsigned WS_USED = 90 * MiB;
constexpr unsigned long long DEADLINE = 30000000ull;

struct Tables { unsigned strip[128]; unsigned char order[5][160]; };
constexpr int arrival(int d, int rb) {
  if (d == 0) return -100;
  if (d < 4) return (rb + 1) * 32;
  return (rb - R1 + 1) * 32 + ((rb - R1 + 1) * 32 <= XA ? 0 : 256);
}
constexpr bool wanted(int type, int d, int rb) {
  if (d == 0) return type == 0 || rb < R1;
  if (d < 4) return rb < R1;
  return rb >= R1 && (type == 0 || d != 3 + type);
}
constexpr Tables make_tables() {
  Tables t{};
  for (int type = 0; type < 5; ++type) {
    int n = 0;
    for (int rb = 0; rb < 32; ++rb)
      for (int d = 0; d < 8; ++d)
        if (wanted(type, d, rb)) {
          int e = d * 32 + rb, a = arrival(d, rb), i = n++;
          while (i > 0 && arrival(t.order[type][i - 1] >> 5, t.order[type][i - 1] & 31) > a) {
            t.order[type][i] = t.order[type][i - 1];
            --i;
          }
          t.order[type][i] = (unsigned char)e;
        }
  }
  for (int k = 0; k < 128; ++k)
    for (int q = 0; q < 4; ++q) t.strip[k] |= (unsigned)t.order[1 + q][k] << (8 * q);
  return t;
}
constexpr bool margins_ok() {
  Tables t = make_tables();
  for (int type = 1; type < 5; ++type)
    for (int k = 0; k < NS; ++k) {
      int a = arrival(t.order[type][k] >> 5, t.order[type][k] & 31);
      if (a < X20) a = X20;
      if (a + 4 * NW > O0 + 2 * k) return false;
    }
  return true;
}
static_assert(margins_ok(), "forwarded strips must not wait for later rounds");
__constant__ const Tables TB = make_tables();

struct Args {
  const float* x;
  const float* w;
  char* ws[8];
  char* out[5];
  int rank;
  int pad;
};

#define WAITV(n) asm volatile("s_waitcnt vmcnt(%0)" ::"i"(n) : "memory")
#define CBAR() asm volatile("" ::: "memory")
constexpr int SYS = 17;
constexpr unsigned OOB = 0xF0000000u;

__device__ __forceinline__ unsigned pk(float a, float b) {
  h16x2 h = {(_Float16)a, (_Float16)b};
  return __builtin_bit_cast(unsigned, h);
}
__device__ __forceinline__ u32x4 pk8(f32x4 a, f32x4 b) {
  return u32x4{pk(a[0], a[1]), pk(a[2], a[3]), pk(b[0], b[1]), pk(b[2], b[3])};
}
__device__ __forceinline__ void st16(char* base, unsigned voff, u32x4 v) {
  asm volatile("global_store_dwordx4 %0, %1, %2 sc0 sc1\n\ts_nop 1" ::"v"(voff), "v"(v), "s"(base) : "memory");
}
__device__ __forceinline__ void put_flag(char* p, unsigned v) {
  __hip_atomic_store((gu32*)p, v, __ATOMIC_RELAXED, __HIP_MEMORY_SCOPE_SYSTEM);
}
__device__ __forceinline__ bool all_ge(u32x4 f, unsigned call) {
  return f[0] >= call && f[1] >= call && f[2] >= call && f[3] >= call;
}
__device__ __forceinline__ unsigned long long now() { return __builtin_amdgcn_s_memrealtime(); }

template <int SLEEP>
__device__ __forceinline__ bool spin4(__amdgpu_buffer_rsrc_t rs, char* ws, unsigned off, bool need, unsigned call,
                                      unsigned long long t0) {
  gu32* tmo = (gu32*)(ws + TMO_OFF);
  for (unsigned it = 0;; ++it) {
    bool ok = true;
    CBAR();
    if (need) ok = all_ge(__builtin_amdgcn_raw_buffer_load_b128(rs, off, 0, SYS), call);
    if (__all(ok)) { CBAR(); return true; }
    if ((it & 15) == 15) {
      if (__hip_atomic_load(tmo, __ATOMIC_RELAXED, __HIP_MEMORY_SCOPE_AGENT) == call || now() - t0 > DEADLINE) {
        __hip_atomic_store(tmo, call, __ATOMIC_RELAXED, __HIP_MEMORY_SCOPE_AGENT);
        return false;
      }
    }
    __builtin_amdgcn_s_sleep(SLEEP);
  }
}

#define STAMP(i) do { } while (0)

struct Round {
  unsigned srcI, dstI;
  unsigned srcC, srcQ;
  unsigned dstC[4];
  int kindI, kindC;
  unsigned flagI, flagC, flagQ;
  unsigned rdyC, rdyQ, rdyL;
};
__device__ __forceinline__ Round make_round(unsigned r, unsigned par, int pw, int rank) {
  Round R;
  const unsigned dummy = DUMMY_OFF + pw * 16384;
  if (r < NI) {
    R.kindI = 1; R.srcI = (r >> 5) * 262144 + (r & 31) * 512; R.dstI = XS_OFF + par + r * 4096; R.flagI = XF_OFF + fl(r);
  } else {
    R.kindI = 0; R.srcI = 0; R.dstI = dummy; R.flagI = 0;
  }
  R.rdyC = 0; R.rdyQ = 0; R.rdyL = 0; R.flagC = 0; R.flagQ = 0;
  if (r >= XA && r < X20) {
    const unsigned p = r - XA;
    R.kindC = 1; R.srcC = WS_OFF + par + p * 4096; R.srcQ = MiB;
#pragma unroll
    for (int q = 0; q < 4; ++q) R.dstC[q] = R.srcC + (4 + q) * MiB;
    R.flagC = WF_OFF + fl(1024 + p); R.flagQ = fl(256); R.rdyC = WF_OFF + fl(p & ~3u); R.rdyQ = fl(256);
  } else if (r < O0) {
    const unsigned p = NI + (r < XA ? r : r - 256);
    R.kindC = 3; R.srcC = (p >> 5) * 262144 + (p & 31) * 512; R.srcQ = 64;
#pragma unroll
    for (int q = 0; q < 4; ++q) R.dstC[q] = XS_OFF + par + p * 4096 + (4 + q) * 4 * MiB;
    R.flagC = XF_OFF + fl(4096 + p); R.flagQ = fl(1024);
  } else if (r < NC) {
    const unsigned k = (r - O0) >> 1, h = (r - O0) & 1;
    R.kindC = 2; R.srcC = OS_OFF + k * 8192 + h * 4096; R.srcQ = MiB;
    const unsigned e4 = TB.strip[k];
#pragma unroll
    for (int q = 0; q < 4; ++q) {
      const unsigned e = (e4 >> (8 * q)) & 255;
      R.dstC[q] = ((rank ^ (e >> 5)) * 512 + (e & 31) * 16 + h * 8) * 2048 + q * 512;
    }
    R.rdyC = OF_OFF + k * 128; R.rdyQ = 16384; R.rdyL = 16;
  } else {
    R.kindC = 0; R.srcC = dummy; R.srcQ = 0;
#pragma unroll
    for (int q = 0; q < 4; ++q) R.dstC[q] = dummy + q * 4096;
  }
  return R;
}

template <int F>
__device__ __forceinline__ void xfer3(u32x4 h, u32x4& a, u32x4& b, unsigned dst, char* b1, char* b2, char* b3, unsigned src, const float* x) {
  asm volatile(
      "global_store_dwordx4 %3, %2, %4 offset:%9 sc0 sc1\n\t"
      "global_store_dwordx4 %3, %2, %5 offset:%9 sc0 sc1\n\t"
      "global_store_dwordx4 %3, %2, %6 offset:%9 sc0 sc1\n\t"
      "global_load_dwordx4 %0, %7, %8 offset:%10\n\t"
      "global_load_dwordx4 %1, %7, %8 offset:%11"
      : "+v"(a), "+v"(b) : "v"(h), "v"(dst), "s"(b1), "s"(b2), "s"(b3), "v"(src), "s"(x), "i"(F * 1024), "i"(F * 128), "i"(F * 128 + 64) : "memory");
}
__device__ __forceinline__ void xfer1(u32x4& v, unsigned dst, char* b, unsigned src, const char* sb) {
  asm volatile(
      "global_store_dwordx4 %1, %0, %2 sc0 sc1\n\t"
      "s_nop 1\n\t"
      "global_load_dwordx4 %0, %3, %4 sc0 sc1"
      : "+v"(v) : "v"(dst), "s"(b), "v"(src), "s"(sb) : "memory");
}
__device__ __forceinline__ void ld16(u32x4& v, unsigned src, const void* sb) {
  asm volatile("global_load_dwordx4 %0, %1, %2 sc0 sc1" : "+v"(v) : "v"(src), "s"(sb) : "memory");
}
__device__ __forceinline__ u32x4 pk8u(u32x4 a, u32x4 b) { return pk8(__builtin_bit_cast(f32x4, a), __builtin_bit_cast(f32x4, b)); }

__device__ __forceinline__ void pusher(const Args& a, int pw, int lane, int wave, gu32* cnt, unsigned long long t0) {
  char* const ws = a.ws[0];
  const unsigned l16 = lane * 16, lout = (lane >> 5) * 2048 + (lane & 31) * 16, lx = (lane & 15) * 16384 + (lane >> 4) * 16;
  u32x4 vA[4], vB[4], vC[4][4], rd = {0, 0, 0, 0};
#pragma unroll
  for (int f = 0; f < 4; ++f) { vA[f] = rd; vB[f] = rd; vC[f][0] = rd; vC[f][1] = rd; vC[f][2] = rd; vC[f][3] = rd; }
  static_assert(XA >= NW, "the first round of every pusher reads x only");
  {
    const unsigned sI = (pw >> 5) * 262144 + (pw & 31) * 512 + lx, sC = sI + R1 * 262144;
#pragma unroll
    for (int f = 0; f < 4; ++f) {
      ld16(vC[f][0], sC + f * 128, a.x); ld16(vC[f][1], sC + f * 128 + 64, a.x);
      ld16(vA[f], sI + f * 128, a.x); ld16(vB[f], sI + f * 128 + 64, a.x);
    }
  }
  const unsigned call = __builtin_amdgcn_readfirstlane(__hip_atomic_load(cnt, __ATOMIC_RELAXED, __HIP_MEMORY_SCOPE_AGENT)) + 1;
  __hip_atomic_store(cnt, call, __ATOMIC_RELAXED, __HIP_MEMORY_SCOPE_AGENT);
  STAMP(0); STAMP(7);
  const unsigned par = (call & 1) * PAR_STRIDE;
  const __amdgpu_buffer_rsrc_t rs = __builtin_amdgcn_make_buffer_rsrc(ws, 0, WS_USED, 0x00020000);
  char* const pl = lane == 1 ? a.ws[1] : lane == 2 ? a.ws[2] : lane == 3 ? a.ws[3] : lane == 4 ? a.ws[4]
                 : lane == 5 ? a.ws[5] : lane == 6 ? a.ws[6] : a.ws[7];
  auto publish = [&](unsigned fI, unsigned fC, unsigned fQ) {
    unsigned off = lane < 4 ? fI + lane * fl(1024) : fC + (lane - 4) * fQ;
    bool on = lane >= 1 && lane < 8 && (lane < 4 ? fI : fC) != 0;
    if (on) put_flag(pl + off, call);
  };
  auto rdy_off = [&](const Round& R) { return R.rdyC + ((lane >> 3) & 3) * R.rdyQ + (lane & 7) * R.rdyL; };
  const unsigned tbase = (call - 1) * NR - 3 * NW;
  unsigned rc = pw, rn = pw + NW, tk = pw + 2 * NW + tbase;
  Round cur = make_round(rc, par, pw, a.rank), nxt = make_round(rn, par, pw, a.rank);
  unsigned pI = 0, pC = 0, pQ = 0;
  const unsigned one = 1, tko = TICK_OFF;
  WAITV(0);
  while (rc < NR) {
#define WAITSTEP(f) asm volatile("s_waitcnt vmcnt(39)" : "+v"(vA[f]), "+v"(vB[f]), "+v"(vC[f][0]), "+v"(vC[f][1]), "+v"(vC[f][2]), "+v"(vC[f][3]) :: "memory")
    WAITSTEP(0);
    asm volatile("" : "+v"(rd), "+v"(tk));
    const bool late = nxt.rdyC != 0 && !__all(all_ge(rd, call));
    const unsigned rnn = __builtin_amdgcn_readfirstlane(tk) - tbase;
    const Round nn = make_round(rnn, par, pw, a.rank);
    ld16(rd, rdy_off(nn), ws);
    tk = one;
    {
      unsigned long long keep;
      asm volatile("s_mov_b64 %1, exec\n\ts_mov_b64 exec, 1\n\tglobal_atomic_add %0, %2, %0, %3 sc0\n\ts_mov_b64 exec, %1"
                   : "+v"(tk), "=&s"(keep) : "v"(tko), "s"(ws) : "memory");
    }
    char* const b1 = cur.kindI ? a.ws[1] + 4 * MiB : ws;
    char* const b2 = cur.kindI ? a.ws[2] + 8 * MiB : ws + 4096;
    char* const b3 = cur.kindI ? a.ws[3] + 12 * MiB : ws + 8192;
    char* const c0 = cur.kindC == 2 ? a.out[1] : cur.kindC ? a.ws[4] : ws;
    char* const c1 = cur.kindC == 2 ? a.out[2] : cur.kindC ? a.ws[5] : ws;
    char* const c2 = cur.kindC == 2 ? a.out[3] : cur.kindC ? a.ws[6] : ws;
    char* const c3 = cur.kindC == 2 ? a.out[4] : cur.kindC ? a.ws[7] : ws;
    const bool curx = cur.kindC == 3, nx = nxt.kindC == 3;
    const char* const sb = nx ? (const char*)a.x : ws;
    const unsigned dstep = cur.kindC == 2 ? 4096 : 1024, sstep = nx ? 128 : 1024;
    const unsigned lc = cur.kindC == 2 ? lout : l16;
    const unsigned dI = cur.dstI + l16, sI = nxt.srcI + lx;
    const unsigned sq = late ? 0 : nxt.srcQ;
    const unsigned s01 = late ? DUMMY_OFF : nxt.srcC + (nx ? lx : l16), s23 = late ? DUMMY_OFF : nxt.srcC + (nx ? 0 : l16 + 2 * sq);
#define STEP(f) { const u32x4 h = pk8u(vA[f], vB[f]), hc = pk8u(vC[f][0], vC[f][1]);                      \
                  vC[f][0] = curx ? hc : vC[f][0]; vC[f][1] = curx ? hc : vC[f][1];                       \
                  vC[f][2] = curx ? hc : vC[f][2]; vC[f][3] = curx ? hc : vC[f][3];                       \
                  xfer1(vC[f][0], cur.dstC[0] + f * dstep + lc, c0, s01 + f * sstep, sb);                 \
                  xfer1(vC[f][1], cur.dstC[1] + f * dstep + lc, c1, s01 + f * sstep + sq, sb);            \
                  xfer3<f>(h, vA[f], vB[f], dI, b1, b2, b3, sI, a.x);                                     \
                  xfer1(vC[f][2], cur.dstC[2] + f * dstep + lc, c2, s23 + f * sstep, sb);                 \
                  xfer1(vC[f][3], cur.dstC[3] + f * dstep + lc, c3, s23 + f * sstep + sq, sb); }
    asm volatile("s_nop 4" :: "s"(b1), "s"(b2), "s"(b3), "s"(c0), "s"(c1), "s"(c2), "s"(c3), "s"(sb), "s"(a.x));
    STEP(0)
    WAITSTEP(1);
    STEP(1)
    WAITSTEP(2);
    STEP(2)
    WAITSTEP(3);
    publish(pI, pC, pQ);
    STEP(3)
#undef STEP
#undef WAITSTEP
    pI = cur.flagI; pC = cur.flagC; pQ = cur.flagQ;
    if (late) {
      WAITV(0);
      publish(pI, pC, pQ);
      pI = pC = 0;
      if (!spin4<1>(rs, ws, rdy_off(nxt), lane < 32, call, t0)) return;
#pragma unroll
      for (int f = 0; f < 4; ++f)
#pragma unroll
        for (int q = 0; q < 4; ++q) ld16(vC[f][q], nxt.srcC + q * nxt.srcQ + f * 1024 + l16, ws);
      ld16(rd, rdy_off(nn), ws);
      WAITV(0);
    }
    cur = nxt; nxt = nn; rc = rn; rn = rnn;
  }
  WAITV(0);
  publish(pI, pC, pQ);
  if (lane >= 4 && lane < 8) put_flag(pl + DF_OFF + fl((lane - 4) * 128 + pw), call);
  STAMP(5);
}

__device__ __forceinline__ void load_x(f32x4 (&s)[8], const float* x, int rb, int slot, int lane) {
  const f32x4* p = (const f32x4*)(x + (size_t)(rb * 16 + (lane & 15)) * 4096 + slot * 128 + (lane >> 4) * 4);
#pragma unroll
  for (int f = 0; f < 4; ++f) { s[2 * f] = p[8 * f]; s[2 * f + 1] = p[8 * f + 4]; }
}

template <int IMM>
__device__ __forceinline__ void ldbuf(u32x4& v, __amdgpu_buffer_rsrc_t rs, unsigned voff, unsigned soff) {
  asm volatile("buffer_load_dwordx4 %0, %1, %2, %3 offen offset:%4 sc0 sc1" : "+v"(v) : "v"(voff), "s"(rs), "s"(soff), "i"(IMM) : "memory");
}
__device__ __forceinline__ void load_block(u32x4 (&A)[16], __amdgpu_buffer_rsrc_t rs, unsigned voff, unsigned soff) {
  asm volatile("s_nop 4");
#define L4(j) ldbuf<0>(A[j], rs, voff, soff + j * 1024); ldbuf<1024>(A[j + 1], rs, voff, soff + j * 1024); \
              ldbuf<2048>(A[j + 2], rs, voff, soff + j * 1024); ldbuf<3072>(A[j + 3], rs, voff, soff + j * 1024);
  L4(0) L4(4) L4(8) L4(12)
#undef L4
}

__device__ __forceinline__ void null_store(__amdgpu_buffer_rsrc_t rs, unsigned v) {
  asm volatile("buffer_store_dword %0, %0, %1, %2 offen" :: "v"(v), "s"(rs), "s"(OOB) : "memory");
}

__global__ void __launch_bounds__(512, 2) ag_gemm(const Args a) {
  extern __shared__ __attribute__((aligned(16))) char smem[];
  const int lane = threadIdx.x & 63;
  const int wave = __builtin_amdgcn_readfirstlane(threadIdx.x >> 6);
  const int xcd = blockIdx.x & 7, slot = blockIdx.x >> 3;
  char* const ws = a.ws[0];
  const unsigned long long t0 = now();
  gu32* const cnt = (gu32*)(ws + CNT_OFF) + blockIdx.x * 8 + wave;

  if (slot >= 28) {
    const int pw = (slot - 28) * 8 + xcd;
    if (wave != 0 || pw >= NW) return;
    pusher(a, pw, lane, wave, cnt, t0);
    return;
  }

  const int id = xcd * 28 + slot;
  const int rep = id < REPO * 16 ? id >> 4 : (id >> 4) - REPO, cg = (id & 15) + (id < REPO * 16 ? 0 : 16);
  const int cb = cg >> 2, g = cg & 3;
  const int type = cb < 4 ? 0 : cb - 3;
  const unsigned l16 = lane * 16;

  unsigned call;
  {
    const int t = wave * 224 + id;
    f32x4 s[8], s2[8];
    if (t < 1024) {
      typedef float f32x2 __attribute__((ext_vector_type(2)));
      const f32x2* p = (const f32x2*)(a.w + (size_t)((t & 63) * 64 + (lane >> 4) * 4) * 512 + (t >> 6) * 32 + (lane & 15) * 2);
#pragma unroll
      for (int j = 0; j < 8; ++j) {
        const f32x2 lo = p[((j & 3) + (j >> 2) * 16) * 256], hi = p[((j & 3) + (j >> 2) * 16 + 32) * 256];
        s[j] = f32x4{lo[0], lo[1], hi[0], hi[1]};
      }
    } else if (t < 1536) {
      __builtin_amdgcn_s_sleep(XDELAY);
      load_x(s, a.x, (t - 1024) >> 4, (t & 15) * 2, lane); load_x(s2, a.x, (t - 1024) >> 4, (t & 15) * 2 + 1, lane);
    }
    call = __builtin_amdgcn_readfirstlane(__hip_atomic_load(cnt, __ATOMIC_RELAXED, __HIP_MEMORY_SCOPE_AGENT)) + 1;
    __hip_atomic_store(cnt, call, __ATOMIC_RELAXED, __HIP_MEMORY_SCOPE_AGENT);
    STAMP(0); STAMP(7);
    const unsigned par = (call & 1) * PAR_STRIDE;
    if (t < 1024) {
      const unsigned o = WS_OFF + par + t * 4096 + l16;
#pragma unroll
      for (int c = 0; c < 4; ++c)
        st16(ws, o + c * 1024, u32x4{pk(s[0][c], s[1][c]), pk(s[2][c], s[3][c]), pk(s[4][c], s[5][c]), pk(s[6][c], s[7][c])});
      WAITV(0);
      if (lane == 0) put_flag(ws + WF_OFF + fl(t), call);
    } else if (t < 1536) {
      const unsigned o = XS_OFF + par + (t - 1024) * 8192 + l16;
#pragma unroll
      for (int f = 0; f < 4; ++f) st16(ws, o + f * 1024, pk8(s[2 * f], s[2 * f + 1]));
#pragma unroll
      for (int f = 0; f < 4; ++f) st16(ws, o + 4096 + f * 1024, pk8(s2[2 * f], s2[2 * f + 1]));
      WAITV(0);
      if (lane < 2) put_flag(ws + XF_OFF + fl((t - 1024) * 2 + lane), call);
    }
  }
  STAMP(1);
  const unsigned par = (call & 1) * PAR_STRIDE;
  const __amdgpu_buffer_rsrc_t rs = __builtin_amdgcn_make_buffer_rsrc(ws, 0, WS_USED, 0x00020000);

  const int nrep = type == 0 ? REPO : REPF;
  const int n = type == 0 ? (NOWN - rep + REPO - 1) / REPO : (NFOR - rep + REPF - 1) / REPF;
  const unsigned ev = TB.order[type][(lane < n ? lane : 0) * nrep + rep];
  const unsigned foff = XF_OFF + fl(ev * 32 + wave * 4);
  const unsigned abase = XS_OFF + par + wave * 16384;
  u32x4 A[16], pf = {0, 0, 0, 0};
#pragma unroll
  for (int j = 0; j < 16; ++j) A[j] = pf;
  bool good = true;
  if (type != 0) {
    good = spin4<SL2>(rs, ws, foff, lane == 0, call, t0);
    load_block(A, rs, l16, abase + (__builtin_amdgcn_readlane(ev, 0) & EMASK) * 131072);
  }
  h16x8 B[16][2];
#pragma unroll
  for (int hf = 0; hf < 2; ++hf) {
    good = spin4<SL2>(rs, ws, WF_OFF + fl(cg * 64 + wave * 8 + hf * 4), lane == 0, call, t0) && good;
    const unsigned so = WS_OFF + par + (cg * 64 + wave * 8) * 4096;
#pragma unroll
    for (int s = hf * 8; s < hf * 8 + 8; ++s)
#pragma unroll
      for (int t = 0; t < 2; ++t)
        B[s][t] = __builtin_bit_cast(h16x8, __builtin_amdgcn_raw_buffer_load_b128(rs, l16 + (s & 1) * 2048 + t * 1024, so + (s >> 1) * 4096, SYS));
  }
  STAMP(2);
  if (type == 0) {
    good = spin4<SL1>(rs, ws, foff, lane == 0, call, t0) && good;
    load_block(A, rs, l16, abase + (__builtin_amdgcn_readlane(ev, 0) & EMASK) * 131072);
  }
  ldbuf<0>(pf, rs, lane >= 1 && lane <= LOOK ? foff : OOB, 0);

  char* const outp = type == 0 ? a.out[0] : type == 1 ? a.out[1] : type == 2 ? a.out[2] : type == 3 ? a.out[3] : a.out[4];
  const __amdgpu_buffer_rsrc_t ro = __builtin_amdgcn_make_buffer_rsrc(outp, 0, 8 * MiB, 0x00020000);
  const unsigned wslot = wave * 2048 + (lane & 15) * 128, wrot = (lane >> 4) * 2 + (lane & 15);
  const unsigned rm = wave * 2 + ((lane >> 3) & 1), rc = lane & 7;
  const unsigned rslot = (lane >> 4) * 2048 + rm * 128 + ((rc + rm) & 7) * 16;
  const unsigned ovo = lane < 16 ? rm * 2048 + rc * 16 : OOB, svo = lane < 16 ? rm * 512 + rc * 16 : OOB;
  unsigned long long mask = 0;
  unsigned pend = 0, pend2 = 0;
  __builtin_amdgcn_s_waitcnt(0x0F70);
  for (int i = 0; i < n; ++i) {
    STAMP(8 + i);
    const unsigned e = __builtin_amdgcn_readlane(ev, i);
    const bool more = i + 1 < n;
    const unsigned nbase = abase + (__builtin_amdgcn_readlane(ev, more ? i + 1 : i) & EMASK) * 131072;
    asm volatile("s_waitcnt vmcnt(17)" : "+v"(pf) :: "memory");
    mask |= __ballot(all_ge(pf, call));
    if (pend) { if (lane == 0) put_flag(ws + pend, call); }
    pend = pend2; pend2 = 0;
    bool have = more && ((mask >> (i + 1)) & 1);
    const unsigned nb = have ? nbase : OOB;
    const unsigned pfo = (lane > i && lane <= i + LOOK && !((mask >> lane) & 1)) ? foff : OOB;
    asm volatile("buffer_load_dwordx4 %0, %1, %2, 0 offen sc0 sc1\n\ts_nop 3" : "+v"(pf) : "v"(pfo), "s"(rs) : "memory");
    f32x4 acc0 = {0, 0, 0, 0}, acc1 = {0, 0, 0, 0};
#define MSTEP(j) asm volatile("s_waitcnt vmcnt(17)" : "+v"(A[j]) :: "memory");                                   \
    acc0 = __builtin_amdgcn_mfma_f32_16x16x32_f16(B[j][0], __builtin_bit_cast(h16x8, A[j]), acc0, 0, 0, 0);      \
    acc1 = __builtin_amdgcn_mfma_f32_16x16x32_f16(B[j][1], __builtin_bit_cast(h16x8, A[j]), acc1, 0, 0, 0);      \
    __builtin_amdgcn_sched_barrier(0);                                                                           \
    ldbuf<(j & 3) * 1024>(A[j], rs, l16, nb + (j >> 2) * 4096);                                                  \
    __builtin_amdgcn_sched_barrier(0);
    MSTEP(0) MSTEP(1) MSTEP(2) MSTEP(3) MSTEP(4) MSTEP(5) MSTEP(6) MSTEP(7)
    MSTEP(8) MSTEP(9) MSTEP(10) MSTEP(11) MSTEP(12) MSTEP(13) MSTEP(14) MSTEP(15)
#undef MSTEP
    if (!have && more) {
      asm volatile("s_waitcnt vmcnt(0)" : "+v"(pf) :: "memory");
      mask |= __ballot(all_ge(pf, call));
      if ((mask >> (i + 1)) & 1) { have = true; load_block(A, rs, l16, nbase); }
    }
    if (!good) { acc0 = __builtin_nanf(""); acc1 = acc0; }
    char* const pb = smem + (i & 1) * 16384;
    *(f32x4*)(pb + wslot + (wrot & 7) * 16) = f32x4{acc0[0], acc1[0], acc0[1], acc1[1]};
    *(f32x4*)(pb + wslot + ((wrot + 1) & 7) * 16) = f32x4{acc0[2], acc1[2], acc0[3], acc1[3]};
    asm volatile("s_waitcnt lgkmcnt(0)\n\ts_barrier" ::: "memory");
    {
      const int k = i * nrep + rep;
      const bool staged = type != 0 && k < NS;
      f32x4 v = *(const f32x4*)(pb + rslot) + *(const f32x4*)(pb + rslot + 8192);
#pragma unroll
      for (int c = 0; c < 4; ++c) {
        const unsigned u = __float_as_uint(v[c]);
        const auto s32 = __builtin_amdgcn_permlane32_swap(u, u, false, false);
        const unsigned u2 = __float_as_uint(__uint_as_float(s32[0]) + __uint_as_float(s32[1]));
        const auto s16 = __builtin_amdgcn_permlane16_swap(u2, u2, false, false);
        v[c] = __uint_as_float(s16[0]) + __uint_as_float(s16[1]);
      }
      const unsigned row = (a.rank ^ (e >> 5)) * 512 + (e & 31) * 16;
      const u32x4 u = __builtin_bit_cast(u32x4, v);
      if (staged) {
        __builtin_amdgcn_raw_buffer_store_b128(u, rs, svo, SOFS(OS_OFF + ((type - 1) * 128 + k) * 8192 + g * 128), SYS);
        pend2 = OF_OFF + (((type - 1) * 128 + k) * 32 + g * 8 + wave) * 4;
      } else {
        if (type == 0) __builtin_amdgcn_raw_buffer_store_b128(u, ro, ovo, SOFS(row * 2048 + (cg & 15) * 128), OAUX);
        else __builtin_amdgcn_raw_buffer_store_b128(u, ro, ovo, SOFS(row * 2048 + (cg & 15) * 128), SYS);
      }
      CBAR();
    }
    if (!have && more) {
      if (pend | pend2) {
        WAITV(0);
        if (lane < 2 && (lane ? pend2 : pend)) put_flag(ws + (lane ? pend2 : pend), call);
        pend = pend2 = 0;
      }
      good = spin4<SL1>(rs, ws, foff, lane == i + 1, call, t0) && good;
      load_block(A, rs, l16, nbase);
      null_store(rs, l16);
    }
  }
  WAITV(0);
  if (lane < 2 && (lane ? pend2 : pend)) put_flag(ws + (lane ? pend2 : pend), call);
  STAMP(3);
  if (type != 0) {
    asm volatile("s_barrier" ::: "memory");
    char* const pw = type == 1 ? a.ws[4] : type == 2 ? a.ws[5] : type == 3 ? a.ws[6] : a.ws[7];
    if (threadIdx.x == 0) put_flag(pw + DF_OFF + fl((type - 1) * 128 + 32 + rep * 4 + g), call);
  } else if (id == 0) {
    asm volatile("s_barrier" ::: "memory");
    if (wave == 0) {
      const unsigned dq = DF_OFF + fl((lane >> 4) * 128 + (lane & 15) * 4);
      if (!spin4<1>(rs, ws, dq, (lane & 15) < NW / 4, call, t0) || !spin4<1>(rs, ws, dq + fl(32), (lane & 15) < REPF, call, t0)) {
        const unsigned e = __builtin_amdgcn_readlane(ev, 0);
        if (lane == 0) put_flag(a.out[0] + (size_t)((a.rank ^ (e >> 5)) * 512 + (e & 31) * 16) * 2048, 0x7fc00000u);
      }
      STAMP(4);
    }
  }
}

}

extern "C" void kernel_launch(int rank, int nranks, ncclComm_t comm, hipStream_t stream,
                              void* const* d_in, const int* in_sizes, int n_in,
                              void* d_out, int out_size, const int* in_dtypes, int out_dtype,
                              void* const* d_out_all, void* const* d_ws_all, size_t ws_bytes) {
  if (nranks != 8 || (unsigned)rank >= 8u || n_in < 2 || ws_bytes < WS_USED || out_size != 4096 * 512 ||
      in_sizes[0] != 512 * 4096 || in_sizes[1] != 4096 * 512) {
    fprintf(stderr, "unexpected problem shape\n");
    return;
  }
  constexpr int LDS_BYTES = 96 * 1024;
  HIP_CHECK(hipFuncSetAttribute((const void*)ag_gemm, hipFuncAttributeMaxDynamicSharedMemorySize, LDS_BYTES));
  Args a;
  a.x = (const float*)d_in[0];
  a.w = (const float*)d_in[1];
  for (int d = 0; d < 8; ++d) a.ws[d] = (char*)d_ws_all[rank ^ d];
  a.out[0] = (char*)d_out;
  for (int q = 0; q < 4; ++q) a.out[1 + q] = (char*)d_out_all[rank ^ (4 + q)];
  a.rank = rank;
  a.pad = 0;
  ag_gemm<<<256, 512, LDS_BYTES, stream>>>(a);
  HIP_CHECK(hipGetLastError());
}
```

```cpp
#include <hip/hip_runtime.h>
#include <rccl/rccl.h>
#include <cstdio>

#define HIP_CHECK(expr) do { hipError_t _e = (expr); if (_e != hipSuccess) { \
    fprintf(stderr, "HIP error %s at %s:%d\n", hipGetErrorString(_e), __FILE__, __LINE__); \
    return; } } while (0)

namespace {

typedef __attribute__((address_space(1))) unsigned gu32;
typedef __attribute__((address_space(3))) unsigned lu32;
typedef unsigned u32x4 __attribute__((ext_vector_type(4)));
typedef float f32x4 __attribute__((ext_vector_type(4)));
typedef float f32x2 __attribute__((ext_vector_type(2)));
typedef _Float16 h16x2 __attribute__((ext_vector_type(2)));
typedef _Float16 h16x8 __attribute__((ext_vector_type(8)));

#define SL1 1
#define SL2 8
#define LOOK 3
#define P_R1 23
#define P_PE 8
#define P_RAWJ 12
#define P_PACE 400
#define P_T0 60
#define P_OSL 1
#define P_GATE 300
#define P_PDLY 24
#define P_FDLY 0
#define P_RO 6
#define P_RF 6, 7, 8, 11
#define P_WIN 18
#define P_CAP 128
constexpr int R1 = P_R1;
constexpr int R2 = 32 - R1;
constexpr int PE = P_PE;
constexpr int RAWJ = P_RAWJ;
constexpr int WIN = P_WIN;
constexpr int REPO = P_RO;
constexpr int REPF[4] = {P_RF};
constexpr int NOWN = 32 + 3 * R1 + 4 * R2;
constexpr int NFOR = 4 * R1 + 3 * R2;
constexpr int NJI = R1 * 8 - PE;
constexpr int JO = 64 + R2 * 8 - PE;
constexpr int NJC = JO + (NFOR + 1) / 2;
static_assert(NFOR <= 128 && NOWN <= 160 && (NOWN + REPO - 1) / REPO <= 52, "");
static_assert(4 * REPO + REPF[0] + REPF[1] + REPF[2] + REPF[3] == 56 && REPF[3] <= 16 && REPF[3] >= 8, "");
static_assert(PE <= 8 && WIN >= 10 && WIN <= 27 && NJI >= 64 && NJC <= 256 && NJI <= 256, "");

constexpr unsigned MiB = 1u << 20;
constexpr unsigned CNT_OFF = 0;
constexpr unsigned TMO_OFF = 16 << 10;
constexpr unsigned TICK_OFF = 20 << 10;
constexpr unsigned RF_OFF = 32 << 10;
constexpr unsigned XF_OFF = 256 << 10;
constexpr unsigned WF_OFF = 512 << 10;
constexpr unsigned PAR_STRIDE = 48 * MiB;
constexpr unsigned XS_OFF = 2 * MiB;
constexpr unsigned WS_OFF = XS_OFF + 32 * MiB;
constexpr unsigned PS_OFF = WS_OFF + 8 * MiB;
constexpr unsigned PSL = 256 + 14 * 1024;
constexpr unsigned OS_OFF = 98 * MiB;
constexpr unsigned STAMP_OFF = 103 * MiB;
constexpr unsigned WS_USED = 104 * MiB;
constexpr unsigned long long DEADLINE = 30000000ull;

struct Tables { unsigned pair[4][64]; unsigned quad[56]; unsigned char order[5][160]; };
constexpr int arrival(int d, int rb) {
  if (d == 0) return rb * 8 >= PE && rb * 8 + 8 <= PE + 64 ? -200 : -100;
  if (d < 4) return (rb + 1) * 8 - PE;
  return (rb - R1 + 1) * 8 <= PE ? 0 : 64 + (rb - R1 + 1) * 8 - PE;
}
constexpr bool wanted(int type, int d, int rb) {
  if (d == 0) return type == 0 || rb < R1;
  if (d < 4) return rb < R1;
  return rb >= R1 && (type == 0 || d != 3 + type);
}
constexpr Tables make_tables() {
  Tables t{};
  for (int type = 0; type < 5; ++type) {
    int n = 0;
    for (int rb = 0; rb < 32; ++rb)
      for (int d = 0; d < 8; ++d)
        if (wanted(type, d, rb)) {
          int e = d * 32 + rb, a = arrival(d, rb), i = n++;
          while (i > 0 && arrival(t.order[type][i - 1] >> 5, t.order[type][i - 1] & 31) > a) {
            t.order[type][i] = t.order[type][i - 1];
            --i;
          }
          t.order[type][i] = (unsigned char)e;
        }
  }
  {
    int used[5] = {0, 0, 0, 0, 0}, nf = 0, i = 0;
    const int tot[5] = {REPO, REPF[0], REPF[1], REPF[2], REPF[3] - 8};
    auto put = [&](unsigned v) { if (i % 7 == 0) { t.quad[i] = 7 | (REPF[3] - 8 + i / 7) << 8 | 255 << 16; ++i; } t.quad[i++] = v; };
    while (i < 56) {
      int best = -1;
      for (int c = 0; c < 5; ++c)
        if (used[c] < tot[c] && (best < 0 || (2 * used[c] + 1) * tot[best] < (2 * used[best] + 1) * tot[c])) best = c;
      if (best == 0) for (int cb = 0; cb < 4; ++cb) put(cb | used[0] << 8 | 255 << 16);
      else put((3 + best) | used[best] << 8 | nf++ << 16);
      ++used[best];
    }
  }
  for (int q = 0; q < 4; ++q)
    for (int m = 0; m < 64; ++m) t.pair[q][m] = t.order[1 + q][2 * m] | (unsigned)t.order[1 + q][2 * m + 1 < NFOR ? 2 * m + 1 : 2 * m] << 8;
  return t;
}
__constant__ const Tables TB = make_tables();

struct Args {
  const float* x;
  const float* w;
  char* ws[8];
  char* out;
  int rank;
  int pad;
};

#define WAITV(n) asm volatile("s_waitcnt vmcnt(%0)" ::"i"(n) : "memory")
#define CBAR() asm volatile("" ::: "memory")
constexpr int SYS = 17;
constexpr unsigned OOB = 0xF0000000u;

__device__ __forceinline__ unsigned pk(float a, float b) {
  h16x2 h = {(_Float16)a, (_Float16)b};
  return __builtin_bit_cast(unsigned, h);
}
__device__ __forceinline__ u32x4 pk8(f32x4 a, f32x4 b) {
  return u32x4{pk(a[0], a[1]), pk(a[2], a[3]), pk(b[0], b[1]), pk(b[2], b[3])};
}
__device__ __forceinline__ void st16(char* base, unsigned voff, u32x4 v) {
  asm volatile("global_store_dwordx4 %0, %1, %2 sc0 sc1\n\ts_nop 1" ::"v"(voff), "v"(v), "s"(base) : "memory");
}
__device__ __forceinline__ void put_flag(char* p, unsigned v) {
  __hip_atomic_store((gu32*)p, v, __ATOMIC_RELAXED, __HIP_MEMORY_SCOPE_SYSTEM);
}
__device__ __forceinline__ bool all_ge(u32x4 f, unsigned call) {
  return f[0] >= call && f[1] >= call && f[2] >= call && f[3] >= call;
}
__device__ __forceinline__ unsigned long long now() { return __builtin_amdgcn_s_memrealtime(); }
__device__ __forceinline__ bool expired(char* ws, unsigned call, unsigned long long t0) {
  gu32* tmo = (gu32*)(ws + TMO_OFF);
  if (__builtin_amdgcn_readfirstlane(__hip_atomic_load(tmo, __ATOMIC_RELAXED, __HIP_MEMORY_SCOPE_AGENT)) != call &&
      (unsigned)(now() - t0) <= (unsigned)DEADLINE) return false;
  __hip_atomic_store(tmo, call, __ATOMIC_RELAXED, __HIP_MEMORY_SCOPE_AGENT);
  return true;
}
template <int SLEEP>
__device__ __forceinline__ bool spin4(__amdgpu_buffer_rsrc_t rs, char* ws, unsigned off, bool need, unsigned call,
                                      unsigned long long t0) {
  for (unsigned it = 0;; ++it) {
    bool ok = true;
    CBAR();
    if (need) ok = all_ge(__builtin_amdgcn_raw_buffer_load_b128(rs, off, 0, SYS), call);
    if (__all(ok)) { CBAR(); return true; }
    if ((it & 15) == 15 && expired(ws, call, t0)) return false;
    __builtin_amdgcn_s_sleep(SLEEP);
  }
}

#define STAMP(i) do { } while (0)
#define STAMPE(i) do { } while (0)

constexpr unsigned HOLE = 0x60606060u;
__device__ __forceinline__ void swap_bits(unsigned& x, unsigned& y, unsigned m) {
  const unsigned nx = (m & y) | (~m & x), ny = (m & x) | (~m & y);
  x = nx; y = ny;
}
__device__ __forceinline__ unsigned rotr(unsigned x, unsigned n) { return __builtin_amdgcn_alignbit(x, x, n); }

__device__ __forceinline__ unsigned pack_slice(const u32x4 (&f)[16], char* ws, unsigned off, char* list, int lane) {
  unsigned sum = 0;
#pragma unroll
  for (int u = 0; u < 8; ++u) sum = __builtin_amdgcn_sad_u8(__builtin_amdgcn_perm(f[2 * u + 1][u & 3], f[2 * u][u & 3], 0x07050301u) & 0x7C7C7C7Cu, 0, sum);
  sum += __builtin_amdgcn_update_dpp(0u, sum, 0x111, 0xf, 0xf, false);
  sum += __builtin_amdgcn_update_dpp(0u, sum, 0x112, 0xf, 0xf, false);
  sum += __builtin_amdgcn_update_dpp(0u, sum, 0x114, 0xf, 0xf, false);
  sum += __builtin_amdgcn_update_dpp(0u, sum, 0x118, 0xf, 0xf, false);
  const int tot = __builtin_amdgcn_readlane(sum, 15) + __builtin_amdgcn_readlane(sum, 31) + __builtin_amdgcn_readlane(sum, 47) + __builtin_amdgcn_readlane(sum, 63);
  const int e0 = (tot - 42189 + 8191) >> 13;
  const unsigned base = e0 < 0 ? 0 : e0 > 24 ? 24 : e0;
  *(unsigned*)(list + lane * 4) = 0;
  unsigned P[8];
#pragma unroll
  for (int h = 0; h < 2; ++h) {
    const unsigned o = off + 256 + h * 7168 + lane * 16;
    u32x4 HH[3];
#pragma unroll
    for (int v = 0; v < 4; ++v) {
      const int u = h * 4 + v;
      u32x4 L;
      unsigned H[4];
#pragma unroll
      for (int j = 0; j < 4; ++j) {
        L[j] = __builtin_amdgcn_perm(f[2 * u + 1][j], f[2 * u][j], 0x06040200u);
        H[j] = __builtin_amdgcn_perm(f[2 * u + 1][j], f[2 * u][j], 0x07050301u) - base * 0x04040404u;
      }
      unsigned g = H[3];
#pragma unroll
      for (int j = 0; j < 3; ++j) { g = rotr(g, 30); swap_bits(H[j], g, HOLE); HH[j][v] = H[j]; }
      st16(ws, o + (3 + v) * 1024, L);
      P[u] = g;
    }
#pragma unroll
    for (int j = 0; j < 3; ++j) st16(ws, o + j * 1024, HH[j]);
  }
  unsigned lo[4], hi[4], cnt = 0;
#pragma unroll
  for (int v = 0; v < 4; ++v) {
    lo[v] = (0x55555555u & rotr(P[2 * v], 1)) | (0xAAAAAAAAu & P[2 * v + 1]);
    hi[v] = (0x55555555u & rotr(P[2 * v], 2)) | (0xAAAAAAAAu & rotr(P[2 * v + 1], 1));
    cnt += __builtin_popcount(lo[v] | hi[v]);
  }
  unsigned pos = cnt;
  pos += __builtin_amdgcn_update_dpp(0u, pos, 0x111, 0xf, 0xf, false);
  pos += __builtin_amdgcn_update_dpp(0u, pos, 0x112, 0xf, 0xf, false);
  pos += __builtin_amdgcn_update_dpp(0u, pos, 0x114, 0xf, 0xf, false);
  pos += __builtin_amdgcn_update_dpp(0u, pos, 0x118, 0xf, 0xf, false);
  pos += __builtin_amdgcn_update_dpp(0u, pos, 0x142, 0xa, 0xf, false);
  pos += __builtin_amdgcn_update_dpp(0u, pos, 0x143, 0xc, 0xf, false);
  if ((unsigned)__builtin_amdgcn_readlane(pos, 63) > P_CAP) return 32;
  char* q = list + (pos - cnt) * 2;
#pragma unroll
  for (int v = 0; v < 4; ++v)
    for (unsigned m = lo[v] | hi[v]; m; m &= m - 1, q += 2) {
      const unsigned s = __builtin_ctz(m);
      *(unsigned short*)q = (unsigned short)((lane * 4 + v) << 7 | s << 2 | ((hi[v] >> s) & 1) << 1 | ((lo[v] >> s) & 1));
    }
  CBAR();
  if (lane < 16) st16(ws, off + lane * 16, *(const u32x4*)(list + lane * 16));
  return base;
}

static_assert(WS_OFF == XS_OFF + 2048 * 16384 && WF_OFF == XF_OFF + 2048 * 128, "");
__device__ __forceinline__ unsigned landing(unsigned j, unsigned d) {
  return j + (d < 4 ? d * 256 + PE : j < 64 ? 2048 + d * 64 : d * 256 + R1 * 8 + PE - 64);
}
__device__ __forceinline__ unsigned strip_dst(unsigned e, unsigned d, int rank) {
  return ((rank ^ d ^ ((e >> 5) & 7)) * 512 + (e & 31) * 16) * 2048 + (d - 4) * 512;
}

constexpr int NSLOT = 5;
constexpr unsigned PLANE_LDS = NSLOT * 16384;
constexpr unsigned CTL_LDS = PLANE_LDS + 6144;

template <int IMM>
__device__ __forceinline__ void dma(unsigned lds, unsigned voff, const char* base) {
  unsigned keep;
  asm volatile("s_mov_b32 %0, m0\n\ts_mov_b32 m0, %1\n\ts_waitcnt vmcnt(%5)\n\tglobal_load_lds_dwordx4 %2, %3 offset:%4 sc0 sc1\n\ts_mov_b32 m0, %0"
               : "=&s"(keep) : "s"(lds), "v"(voff), "s"(base), "i"(IMM), "i"(WIN - 1) : "memory");
}
__device__ __forceinline__ void dma_masked(unsigned lds, unsigned voff, const char* base, unsigned long long lanes) {
  unsigned keep;
  asm volatile("s_mov_b32 %0, m0\n\ts_mov_b32 m0, %1\n\ts_mov_b64 exec, %4\n\ts_waitcnt vmcnt(%5)\n\t"
               "global_load_lds_dwordx4 %2, %3 sc0 sc1\n\ts_mov_b64 exec, -1\n\ts_mov_b32 m0, %0"
               : "=&s"(keep) : "s"(lds), "v"(voff), "s"(base), "s"(lanes), "i"(WIN - 1) : "memory");
}

__device__ __forceinline__ void loader(const Args& a, char* smem, unsigned d, unsigned w, int lane, int wave, unsigned call,
                                       unsigned long long t0) {
  char* const ws = a.ws[0];
  const char* const peer = d == 1 ? a.ws[1] : d == 2 ? a.ws[2] : d == 3 ? a.ws[3] : d == 4 ? a.ws[4] : d == 5 ? a.ws[5] : d == 6 ? a.ws[6] : a.ws[7];
  const unsigned l16 = lane * 16, par = (call & 1) * PAR_STRIDE, nj = d < 4 ? NJI : NJC, jo = d < 4 ? 256 : JO, jw = d < 4 ? 0 : 64;
  const unsigned rf = RF_OFF + ((call & 1) * 8 + d) * 1024, tko = TICK_OFF + d * 128;
  const unsigned lds0 = (unsigned)(size_t)(lu32*)smem;
  volatile lu32* const ctl = (volatile lu32*)(smem + CTL_LDS);
  const unsigned xs = d < 4 ? PE : R1 * 8 + PE - 64;
  const unsigned rawX = par + XS_OFF + xs * 16384, rawW = par + WS_OFF + (d - 4) * 64 * 16384, rawO = OS_OFF + ((d - 4) * 64 - JO) * 16384;
  const unsigned pakX = par + PS_OFF + xs * PSL, pakW = par + PS_OFF + d * 64 * PSL;
  const unsigned thrS = call << 6, thrO = 8 * ((call + 1) >> 1);
  auto words = [&](unsigned j) {
    const unsigned k = ((j & 240) + (lane & 31)) & 255;
    return (k >= jo && k < nj ? peer : ws) + rf + k * 4;
  };
  int landed = 0;
  auto publish = [&](int f) { if (f > landed) { landed = f; ctl[0] = f; } };
  const unsigned tbase = (call - 1) * nj - 4;
  unsigned j = w, tk = 0, got = 0, info = 0, freed = 0;
  int n = 0, slot = 0, lap = 0;
  for (;;) {
    if (j >= nj || info < (j >= jo ? thrO : thrS)) {
      unsigned fresh = 0;
      asm volatile("global_load_dword %0, %1, off sc0 sc1\n\ts_waitcnt vmcnt(9)" : "+v"(fresh) : "v"(words(j)) : "memory");
      publish(2 * n - 1);
      WAITV(1); publish(2 * n);
      asm volatile("s_waitcnt vmcnt(0)" : "+v"(fresh) :: "memory");
      if (j >= nj) break;
      STAMP(8);
      info = __builtin_amdgcn_readlane(fresh, j & 15);
      bool ok = true;
      for (unsigned it = 0; info < (j >= jo ? thrO : thrS); ++it) {
        if ((it & 15) == 15 && expired(ws, call, t0)) { ok = false; break; }
        info = __builtin_amdgcn_readfirstlane(__hip_atomic_load((const gu32*)((j >= jo ? peer : ws) + rf + j * 4), __ATOMIC_RELAXED, __HIP_MEMORY_SCOPE_SYSTEM));
      }
      CBAR();
      if (!ok) {
        if (j >= jo) put_flag(a.out + strip_dst(TB.pair[d & 3][(j - JO) & 63], d, a.rank), 0x7fc00000u);
        break;
      }
      if (n == 0) while ((unsigned)(now() - t0) < P_GATE) __builtin_amdgcn_s_sleep(1);
      STAMP(9);
    }
    if (j >= jo) info = 32;
    const bool raw = info & 32;
    const unsigned src = (raw ? (j >= jo ? rawO : j < jw ? rawW : rawX) : j < jw ? pakW : pakX) + j * (raw ? 16384 : PSL);
    while ((int)__builtin_amdgcn_readfirstlane(freed) < 2 * lap) freed = ctl[4 + slot];
    *(volatile __attribute__((address_space(3))) unsigned long long*)(ctl + 16 + 2 * slot) = (unsigned long long)info << 32 | j;
    const unsigned lds = lds0 + slot * 16384;
    const unsigned v0 = src + l16 - (raw ? 0 : 768), v1 = v0 + (!raw ? 7168 : (NFOR & 1) && j >= jo && j == NJC - 1 ? 0 : 8192);
    const char* const b8 = raw ? peer : ws;
#define OP(i) if (i == (9 + WIN) % 18) { WAITV(WIN - 1); publish(2 * (n - (9 + WIN) / 18) + 1); } \
              if (i == (17 + WIN) % 18) { WAITV(WIN - 1); publish(2 * (n - (17 + WIN) / 18) + 2); }
    tk = 1;
    OP(0) asm volatile("s_mov_b64 exec, 1\n\ts_nop 3\n\ts_waitcnt vmcnt(%3)\n\tglobal_atomic_add %0, %1, %0, %2 sc0\n\ts_mov_b64 exec, -1" : "+v"(tk) : "v"(tko), "s"(ws), "i"(WIN - 1), "s"(peer), "s"(b8) : "memory");
    OP(1) asm volatile("s_waitcnt vmcnt(%2)\n\tglobal_load_dword %0, %1, off sc0 sc1" : "+v"(got) : "v"(words(j)), "i"(WIN - 1) : "memory");
    OP(2) dma_masked(lds, src + l16, peer, raw ? ~0ull : 0xFFFFull);
    OP(3) dma<1024>(lds, v0, peer);
    OP(4) dma<2048>(lds, v0, peer);
    OP(5) dma<3072>(lds, v0, peer);
    OP(6) dma<0>(lds + 4096, v0 + 4096, peer);
    OP(7) dma<1024>(lds + 4096, v0 + 4096, peer);
    OP(8) dma<2048>(lds + 4096, v0 + 4096, peer);
    OP(9) dma<3072>(lds + 4096, v0 + 4096, peer);
    OP(10) dma_masked(lds + 8192, raw ? v1 : CNT_OFF, b8, raw ? ~0ull : 1ull);
    OP(11) dma<1024>(lds + 8192, v1, peer);
    OP(12) dma<2048>(lds + 8192, v1, peer);
    OP(13) dma<3072>(lds + 8192, v1, peer);
    OP(14) dma<0>(lds + 12288, v1 + 4096, peer);
    OP(15) dma<1024>(lds + 12288, v1 + 4096, peer);
    OP(16) dma<2048>(lds + 12288, v1 + 4096, peer);
    OP(17) dma<3072>(lds + 12288, v1 + 4096, peer);
#undef OP
    ++n;
    if (++slot == NSLOT) { slot = 0; ++lap; }
    freed = ctl[4 + slot];
    asm volatile(".Lag_last:\n\ts_waitcnt vmcnt(16)" : "+v"(got), "+v"(tk) :: "memory");
    publish(2 * n - 2);
    const unsigned jc = j;
    j = __builtin_amdgcn_readfirstlane(tk) - tbase;
    info = j - (jc & 240) < 32 ? __builtin_amdgcn_readlane(got, (j - (jc & 240)) & 31) : 0;
  }
  ctl[1] = n + 1;
  STAMP(10);
}

__device__ __forceinline__ void unpacker(const Args& a, char* smem, unsigned d, int lane, int wave, unsigned call, unsigned long long t0) {
  char* const ws = a.ws[0];
  const unsigned h = (wave - 1) & 1, l16 = lane * 16, lout = (lane >> 5) * 2048 + (lane & 31) * 16;
  volatile lu32* const ctl = (volatile lu32*)(smem + CTL_LDS);
  char* const plane = smem + PLANE_LDS + (wave - 1) * 1024;
  const u32x4 zero = {0, 0, 0, 0};
  *(u32x4*)(plane + l16) = zero;
  const unsigned odst = strip_dst(TB.pair[d & 3][lane] >> (8 * h), d, a.rank);
  for (unsigned n = (wave - 1) >> 1;; n += 3) {
    bool over = false;
    for (;;) {
      const unsigned all = __builtin_amdgcn_readfirstlane(ctl[1]), landed = __builtin_amdgcn_readfirstlane(ctl[0]);
      if (landed > 2 * n + h) break;
      if (all) { over = true; break; }
      __builtin_amdgcn_s_sleep(1);
    }
    if (over) break;
    CBAR();
    const unsigned slot = n % NSLOT;
    const char* const p = smem + slot * 16384 + h * 8192 + l16;
    const unsigned long long job = *(const unsigned long long*)(smem + CTL_LDS + 64 + slot * 8);
    const unsigned lst = *(const unsigned*)(smem + slot * 16384 + lane * 4);
    u32x4 r[8];
#pragma unroll
    for (int i = 0; i < 8; ++i) r[i] = *(const u32x4*)(p + i * 1024);
    const unsigned j = __builtin_amdgcn_readfirstlane((unsigned)job), info = __builtin_amdgcn_readfirstlane((unsigned)(job >> 32));
    const bool strip = d >= 4 && j >= JO;
    const unsigned flag = strip ? 0 : XF_OFF + landing(j, d) * 128;
    const unsigned dst = strip ? __builtin_amdgcn_readlane(odst, (j - JO) & 63) : (call & 1) * PAR_STRIDE + XS_OFF + landing(j, d) * 16384 + h * 8192;
    if (!(info & 32)) {
#pragma unroll
      for (int c = 0; c < 2; ++c) {
        const unsigned e = c ? lst >> 16 : lst & 0xffff;
        if (((e >> 8) & 1) == h)
          __hip_atomic_fetch_or((lu32*)(plane + ((e >> 9) * 16 | ((e >> 4) & 8) | (e & 4))), rotr((e & 3) << ((e >> 2) & 30), 31),
                                __ATOMIC_RELAXED, __HIP_MEMORY_SCOPE_WORKGROUP);
      }
      CBAR();
      r[0] = *(const u32x4*)(plane + l16);
      CBAR();
      *(u32x4*)(plane + l16) = zero;
    }
    asm volatile("s_waitcnt lgkmcnt(0)" : "+v"(r[0]), "+v"(r[1]), "+v"(r[2]), "+v"(r[3]), "+v"(r[4]), "+v"(r[5]), "+v"(r[6]), "+v"(r[7]) :: "memory");
    if (lane == 0) __hip_atomic_fetch_add((lu32*)(ctl + 4 + slot), 1u, __ATOMIC_RELAXED, __HIP_MEMORY_SCOPE_WORKGROUP);
    if (info & 32) {
      char* const db = flag ? ws : a.out;
      const unsigned o = dst + (flag ? l16 : lout), step = flag ? 1024 : 4096;
      asm volatile("s_nop 4" :: "s"(db));
#pragma unroll
      for (int i = 0; i < 8; ++i) st16(db, o + i * step, r[i]);
    } else {
      const unsigned base = (info & 31) * 0x04040404u;
#pragma unroll
      for (int v = 0; v < 4; ++v) {
        unsigned h0 = r[1][v], h1 = r[2][v], h2 = r[3][v], g = r[0][v];
        swap_bits(h2, g, HOLE); g = rotr(g, 2); swap_bits(h1, g, HOLE); g = rotr(g, 2); swap_bits(h0, g, HOLE); g = rotr(g, 2);
        h0 += base; h1 += base; h2 += base; g += base;
        const u32x4 L = r[4 + v];
        st16(ws, dst + l16 + v * 2048, u32x4{__builtin_amdgcn_perm(h0, L[0], 0x05010400u), __builtin_amdgcn_perm(h1, L[1], 0x05010400u),
                                             __builtin_amdgcn_perm(h2, L[2], 0x05010400u), __builtin_amdgcn_perm(g, L[3], 0x05010400u)});
        st16(ws, dst + l16 + v * 2048 + 1024, u32x4{__builtin_amdgcn_perm(h0, L[0], 0x07030602u), __builtin_amdgcn_perm(h1, L[1], 0x07030602u),
                                                    __builtin_amdgcn_perm(h2, L[2], 0x07030602u), __builtin_amdgcn_perm(g, L[3], 0x07030602u)});
      }
    }
    if (flag) {
      WAITV(0);
      if (lane < 2) put_flag(ws + flag + h * 8 + lane * 4, call);
    }
  }
  STAMPE(d < 4 ? 4 : 5);
}

template <int N>
__device__ __forceinline__ void load_x(f32x4 (&s)[2 * N], const float* x, int rb, int k0, int lane) {
  const f32x4* p = (const f32x4*)(x + (size_t)(rb * 16 + (lane & 15)) * 4096 + k0 + (lane >> 4) * 4);
#pragma unroll
  for (int f = 0; f < N; ++f) { s[2 * f] = p[8 * f]; s[2 * f + 1] = p[8 * f + 4]; }
}
template <int IMM>
__device__ __forceinline__ void ldbuf(u32x4& v, __amdgpu_buffer_rsrc_t rs, unsigned voff, unsigned soff) {
  asm volatile("buffer_load_dwordx4 %0, %1, %2, %3 offen offset:%4 sc0 sc1" : "+v"(v) : "v"(voff), "s"(rs), "s"(soff), "i"(IMM) : "memory");
}
__device__ __forceinline__ void load_block(u32x4 (&A)[16], __amdgpu_buffer_rsrc_t rs, unsigned voff, unsigned soff) {
  asm volatile("s_nop 4");
#define L4(j) ldbuf<0>(A[j], rs, voff, soff + j * 1024); ldbuf<1024>(A[j + 1], rs, voff, soff + j * 1024); \
              ldbuf<2048>(A[j + 2], rs, voff, soff + j * 1024); ldbuf<3072>(A[j + 3], rs, voff, soff + j * 1024);
  L4(0) L4(4) L4(8) L4(12)
#undef L4
}
__device__ __forceinline__ void stbuf(u32x4 v, __amdgpu_buffer_rsrc_t rs, unsigned voff, unsigned soff) {
  asm volatile("s_nop 4\n\tbuffer_store_dwordx4 %0, %1, %2, %3 offen sc0 sc1\n\ts_nop 1" :: "v"(v), "v"(voff), "s"(rs), "s"(soff) : "memory");
}
__device__ __forceinline__ void null_store(__amdgpu_buffer_rsrc_t rs, unsigned v) {
  asm volatile("buffer_store_dword %0, %0, %1, %2 offen" :: "v"(v), "s"(rs), "s"(OOB) : "memory");
}
__device__ __forceinline__ unsigned next_call(gu32* cnt) {
  const unsigned call = __builtin_amdgcn_readfirstlane(__hip_atomic_load(cnt, __ATOMIC_RELAXED, __HIP_MEMORY_SCOPE_AGENT)) + 1;
  __hip_atomic_store(cnt, call, __ATOMIC_RELAXED, __HIP_MEMORY_SCOPE_AGENT);
  return call;
}

__global__ void __launch_bounds__(512, 2) ag_gemm(const Args a) {
  extern __shared__ __attribute__((aligned(16))) char smem[];
  {
    unsigned long long pc;
    unsigned off = (blockIdx.x >> 3) * 1024 + (threadIdx.x >> 1) * 4;
    asm volatile("s_getpc_b64 %0\n.Lag_first:\n\tv_min_u32 %1, .Lag_last-.Lag_first, %1\n\tglobal_load_dword v255, %1, %0"
                 : "=&s"(pc), "+v"(off) :: "v255", "memory");
  }
  const int lane = threadIdx.x & 63;
  const int wave = __builtin_amdgcn_readfirstlane(threadIdx.x >> 6);
  const int xcd = blockIdx.x & 7, slot = blockIdx.x >> 3;
  char* const ws = a.ws[0];
  const unsigned long long t0 = now();
  gu32* const cnt = (gu32*)(ws + CNT_OFF) + blockIdx.x * 8 + wave;
  const unsigned l16 = lane * 16;
  const __amdgpu_buffer_rsrc_t rs = __builtin_amdgcn_make_buffer_rsrc(ws, 0, WS_USED, 0x00020000);
  auto lane_peer = [&] {
    unsigned lo = 0, hi = 0;
#pragma unroll
    for (int d = 1; d < 8; ++d) {
      unsigned l = (unsigned)(size_t)a.ws[d], h = (unsigned)((size_t)a.ws[d] >> 32);
      asm volatile("" : "+s"(l), "+s"(h));
      lo = lane == d ? l : lo;
      hi = lane == d ? h : hi;
    }
    return (char*)((size_t)hi << 32 | lo);
  };

  constexpr int NU = 5 * RAWJ;
  static_assert(NU <= 96, "");
  unsigned call;
  if (blockIdx.x < 32) {
    const bool on = wave < 4 && blockIdx.x < PE * 4;
    f32x4 s[2], s2[2];
    if (on) { load_x<1>(s, a.x, 0, blockIdx.x * 128 + wave * 32, lane); load_x<1>(s2, a.x, R1, blockIdx.x * 128 + wave * 32, lane); }
    call = next_call(cnt);
    STAMP(0); STAMP(7);
    if (on) {
      const unsigned o1 = XS_OFF + (call & 1) * PAR_STRIDE + blockIdx.x * 4096 + wave * 1024 + l16, o2 = o1 + R1 * 131072;
      const u32x4 h1 = pk8(s[0], s[1]), h2 = pk8(s2[0], s2[1]);
      for (int i = 1; i < wave; ++i) __builtin_amdgcn_s_sleep(P_PDLY);
      st16(a.ws[4] + 16 * MiB, o2, h2); st16(a.ws[1] + 4 * MiB, o1, h1);
      st16(a.ws[5] + 20 * MiB, o2, h2); st16(a.ws[2] + 8 * MiB, o1, h1);
      st16(a.ws[6] + 24 * MiB, o2, h2); st16(a.ws[3] + 12 * MiB, o1, h1);
      st16(a.ws[7] + 28 * MiB, o2, h2);
    }
    char* const lp = lane_peer();
    asm volatile("s_waitcnt vmcnt(0)\n\ts_barrier" ::: "memory");
    if (on && wave == 0 && lane >= 1 && lane < 8)
      put_flag(lp + XF_OFF + (lane * 256 + (lane >= 4 ? R1 * 8 : 0) + (blockIdx.x >> 2)) * 128 + (blockIdx.x & 3) * 4, call);
    STAMP(6);
  } else if (blockIdx.x < 32 + NU) {
    const int k = (blockIdx.x - 32) / 5, i = blockIdx.x - 32 - k * 5;
    u32x4 f0, f1;
    if (i == 0) {
      f32x4 s[4];
      load_x<2>(s, a.x, (PE + k) >> 3, ((PE + k) & 7) * 512 + wave * 64, lane);
      call = next_call(cnt);
      f0 = pk8(s[0], s[1]); f1 = pk8(s[2], s[3]);
    } else {
      const f32x2* p = (const f32x2*)(a.w + (size_t)((k & 15) * 256 + wave * 32 + (lane >> 4) * 4) * 512 + ((i - 1) * 4 + (k >> 4)) * 32 + (lane & 15) * 2);
      f32x2 s[8];
#pragma unroll
      for (int j = 0; j < 8; ++j) s[j] = p[((j & 3) + (j >> 2) * 16) * 256];
      call = next_call(cnt);
      f0 = u32x4{pk(s[0][0], s[1][0]), pk(s[2][0], s[3][0]), pk(s[4][0], s[5][0]), pk(s[6][0], s[7][0])};
      f1 = u32x4{pk(s[0][1], s[1][1]), pk(s[2][1], s[3][1]), pk(s[4][1], s[5][1]), pk(s[6][1], s[7][1])};
    }
    STAMP(11); STAMP(7);
    const unsigned gs = i ? 2048 + (i - 1) * 64 + k : PE + k;
    const unsigned o = XS_OFF + (call & 1) * PAR_STRIDE + gs * 16384 + wave * 2048 + l16;
    st16(ws, o, f0); st16(ws, o + 1024, f1);
    char* const lp = lane_peer();
    asm volatile("s_waitcnt vmcnt(0)\n\ts_barrier" ::: "memory");
    if (wave == 7) {
      if (P_FDLY) while ((unsigned)(now() - t0) < P_FDLY) __builtin_amdgcn_s_sleep(1);
      if (i ? lane == 3 + i : lane >= 1 && lane < 4) put_flag(lp + RF_OFF + ((call & 1) * 8 + lane) * 1024 + k * 4, call << 6 | 32);
      if (lane < 4) put_flag(ws + XF_OFF + gs * 128 + lane * 4, call);
    }
    STAMP(12);
  } else {
    call = next_call(cnt);
    STAMP(7);
  }

  if (slot >= 28) {
    const int sw = blockIdx.x - 224;
    STAMP(0);
    if (threadIdx.x < 16) *(unsigned*)(smem + CTL_LDS + lane * 4) = 0;
    asm volatile("s_waitcnt lgkmcnt(0)\n\ts_barrier" ::: "memory");
    if (sw < 28) {
      if (wave == 0) loader(a, smem, 1 + sw % 7, sw / 7, lane, wave, call, t0);
      else if (wave < 7) unpacker(a, smem, 1 + sw % 7, lane, wave, call, t0);
    }
    return;
  }

  const int id = xcd * 28 + slot;
  const unsigned role = TB.quad[id >> 2];
  const bool isown = (role & 255) < 4;
  const int rep = (role >> 8) & 255, g = isown ? id & 3 : (role & 255) - 4;
  const int type = isown ? 0 : 1 + (id & 3), cg = isown ? (role & 255) * 4 + g : 12 + type * 4 + g;
  {
    constexpr int NF = 192 - 16 * REPO;
    const int t = NU + (slot < 4 ? 512 : wave < 2 ? wave * 192 + 223 - (int)blockIdx.x : 384 + (wave - 2) * NF + (role >> 16) * 4 + (id & 3));
    if (t < 512) {
      int sl;
      if (t < 320) { const int k = t / 5, i = t - k * 5; sl = i ? 256 + (i - 1) * 64 + k : PE + k; }
      else if (t < 320 + 2 * PE) sl = t - 320 < PE ? t - 320 : R1 * 8 + t - 320 - PE;
      else { sl = t - 2 * PE - 320 + PE + 64; if (sl >= R1 * 8) sl += PE; }
      while (now() - t0 < P_T0 + ((unsigned)((t - NU) * P_PACE) >> 9)) __builtin_amdgcn_s_sleep(2);
      u32x4 f[16];
      if (sl < 256) {
        f32x4 s[32];
        load_x<16>(s, a.x, sl >> 3, (sl & 7) * 512, lane);
#pragma unroll
        for (int i = 0; i < 16; ++i) f[i] = pk8(s[2 * i], s[2 * i + 1]);
      } else {
        const f32x2* p = (const f32x2*)(a.w + (size_t)((sl & 15) * 256 + (lane >> 4) * 4) * 512 + ((sl - 256) >> 4) * 32 + (lane & 15) * 2);
        f32x2 s[64];
#pragma unroll
        for (int i = 0; i < 64; ++i) s[i] = p[((i >> 3) * 32 + (i & 3) + ((i >> 2) & 1) * 16) * 256];
#pragma unroll
        for (int i = 0; i < 16; ++i) {
          const int b = (i >> 1) * 8, c = i & 1;
          f[i] = u32x4{pk(s[b][c], s[b + 1][c]), pk(s[b + 2][c], s[b + 3][c]), pk(s[b + 4][c], s[b + 5][c]), pk(s[b + 6][c], s[b + 7][c])};
        }
      }
      STAMP(0);
      const unsigned par = (call & 1) * PAR_STRIDE, gs = sl < 256 ? sl : 2048 - 256 + sl;
#pragma unroll
      for (int i = 0; i < 16; ++i) st16(ws, par + XS_OFF + gs * 16384 + l16 + i * 1024, f[i]);
      const int job = sl >= 256 ? sl & 63 : sl < R1 * 8 ? sl - PE : sl - R1 * 8 < PE ? -1 : sl - R1 * 8 - PE + 64;
      const bool mine = sl >= 256 ? lane == sl >> 6 : sl < R1 * 8 ? lane >= 1 && lane < 4 : lane >= 4 && lane < 8;
      unsigned info = 32;
      STAMP(6);
      if (job >= 0) info = pack_slice(f, ws, par + PS_OFF + sl * PSL, smem + 36864 + wave * 256, lane);
      STAMP(5);
      char* const lp = lane_peer();
      WAITV(0);
      if (job >= 0 && mine) put_flag(lp + RF_OFF + ((call & 1) * 8 + lane) * 1024 + job * 4, call << 6 | info);
      if (lane < 4) put_flag(ws + XF_OFF + gs * 128 + lane * 4, call);
    }
  }
  STAMP(1);
  const unsigned par = (call & 1) * PAR_STRIDE;
  if (threadIdx.x < 64) *(unsigned*)(smem + 32768 + lane * 4) = 0;
  asm volatile("s_waitcnt lgkmcnt(0)\n\ts_barrier" ::: "memory");

  const int nrep = isown ? REPO : g == 0 ? REPF[0] : g == 1 ? REPF[1] : g == 2 ? REPF[2] : REPF[3];
  const int n = __popcll(__ballot(lane * nrep + rep < (isown ? NOWN : NFOR)));
  const unsigned ev = TB.order[type][(lane < n ? lane : 0) * nrep + rep];
  const unsigned foff = XF_OFF + (ev * 8 + wave) * 128;
  const unsigned abase = XS_OFF + par + wave * 16384;
  u32x4 A[16], pf = {0, 0, 0, 0};
#pragma unroll
  for (int j = 0; j < 16; ++j) A[j] = pf;
  bool good = true;
  if (type != 0) {
    good = spin4<SL2>(rs, ws, foff, lane == 0, call, t0);
    load_block(A, rs, l16, abase + __builtin_amdgcn_readlane(ev, 0) * 131072);
  }
  h16x8 B[16][2];
#pragma unroll
  for (int hf = 0; hf < 2; ++hf) {
    good = spin4<SL2>(rs, ws, WF_OFF + (cg * 16 + wave * 2 + hf) * 128, lane == 0, call, t0) && good;
    const unsigned so = WS_OFF + par + (cg * 16 + wave * 2) * 16384;
#pragma unroll
    for (int s = hf * 8; s < hf * 8 + 8; ++s)
#pragma unroll
      for (int t = 0; t < 2; ++t)
        B[s][t] = __builtin_bit_cast(h16x8, __builtin_amdgcn_raw_buffer_load_b128(rs, l16 + (s & 1) * 2048 + t * 1024, so + (s >> 1) * 4096, SYS));
  }
  STAMP(2);
  if (type == 0) {
    good = spin4<SL1>(rs, ws, foff, lane == 0, call, t0) && good;
    load_block(A, rs, l16, abase + __builtin_amdgcn_readlane(ev, 0) * 131072);
  }
  ldbuf<0>(pf, rs, lane >= 1 && lane <= LOOK ? foff : OOB, 0);

  const __amdgpu_buffer_rsrc_t ro = __builtin_amdgcn_make_buffer_rsrc(isown ? a.out : ws + OS_OFF, 0, isown ? 8 * MiB : 4 * MiB, 0x00020000);
  const unsigned wslot = wave * 2048 + (lane & 15) * 128, wrot = (lane >> 4) * 2 + (lane & 15);
  const unsigned rm = wave * 2 + ((lane >> 3) & 1), rc = lane & 7;
  const unsigned rslot = (lane >> 4) * 2048 + rm * 128 + ((rc + rm) & 7) * 16;
  const unsigned ovo = lane < 16 ? rm * (isown ? 2048 : 512) + rc * 16 : OOB;
  lu32* const lcnt = (lu32*)(smem + 32768);
  auto reduce = [&](int i) -> unsigned {
    const unsigned goal = 8 * ((i >> 1) + 1);
    while (__hip_atomic_load(lcnt + (i & 1) * 32, __ATOMIC_ACQUIRE, __HIP_MEMORY_SCOPE_WORKGROUP) < goal) __builtin_amdgcn_s_sleep(1);
    const char* const pb = smem + (i & 1) * 16384;
    f32x4 v = *(const f32x4*)(pb + rslot) + *(const f32x4*)(pb + rslot + 8192);
#pragma unroll
    for (int c = 0; c < 4; ++c) {
      const unsigned u = __float_as_uint(v[c]);
      const auto s32 = __builtin_amdgcn_permlane32_swap(u, u, false, false);
      const unsigned u2 = __float_as_uint(__uint_as_float(s32[0]) + __uint_as_float(s32[1]));
      const auto s16 = __builtin_amdgcn_permlane16_swap(u2, u2, false, false);
      v[c] = __uint_as_float(s16[0]) + __uint_as_float(s16[1]);
    }
    const unsigned e = __builtin_amdgcn_readlane(ev, i);
    const unsigned so = isown ? ((a.rank ^ (e >> 5)) * 512 + (e & 31) * 16) * 2048 + cg * 128
                              : ((type - 1) * 128 + i * nrep + rep) * 8192 + g * 128;
    stbuf(__builtin_bit_cast(u32x4, v), ro, ovo, so);
    return isown ? 0 : i + 1;
  };
  auto raise2 = [&](unsigned f0, unsigned f1) {
    const unsigned f = lane ? f1 : f0;
    if (lane < 2 && f) {
      if ((__hip_atomic_fetch_add(lcnt + 8 + (f & 3), 1u, __ATOMIC_RELAXED, __HIP_MEMORY_SCOPE_WORKGROUP) & 7) == 7)
      {
        const unsigned k = (f - 1) * nrep + rep;
        __hip_atomic_fetch_add((gu32*)(ws + RF_OFF + ((call & 1) * 8 + 3 + type) * 1024 + (JO + (k >> 1)) * 4), (k ^ 1) < NFOR ? 1u : 2u,
                               __ATOMIC_RELAXED, __HIP_MEMORY_SCOPE_SYSTEM);
      }
    }
  };
  unsigned long long mask = 0;
  unsigned pend = 0, fresh = 0;
  bool owed = false;
  __builtin_amdgcn_s_waitcnt(0x0F70);
  for (int i = 0; i < n; ++i) {
    STAMP(8 + i);
    const bool more = i + 1 < n;
    const unsigned nbase = abase + __builtin_amdgcn_readlane(ev, more ? i + 1 : i) * 131072;
    asm volatile("s_waitcnt vmcnt(17)" : "+v"(pf) :: "memory");
    mask |= __ballot(all_ge(pf, call));
    const unsigned hv = (unsigned)(mask >> (i + 1)) & (more ? 1u : 0u);
    bool have = hv != 0;
    const unsigned pfo = (lane > i && lane <= i + LOOK && !((mask >> lane) & 1)) ? foff : OOB;
    asm volatile("buffer_load_dwordx4 %0, %1, %2, 0 offen sc0 sc1\n\ts_nop 3" : "+v"(pf) : "v"(pfo), "s"(rs) : "memory");
    f32x4 acc0 = {0, 0, 0, 0}, acc1 = {0, 0, 0, 0};
#define MS(j) asm volatile("s_cmp_lg_u32 %1, 0\n\ts_cbranch_scc1 1f\n\ts_waitcnt vmcnt(%2)\n1:\n\ts_waitcnt vmcnt(17)"       \
                           : "+v"(A[j]) : "s"(hv), "i"(16 - j) : "memory", "scc");                               \
    acc0 = __builtin_amdgcn_mfma_f32_16x16x32_f16(B[j][0], __builtin_bit_cast(h16x8, A[j]), acc0, 0, 0, 0);      \
    acc1 = __builtin_amdgcn_mfma_f32_16x16x32_f16(B[j][1], __builtin_bit_cast(h16x8, A[j]), acc1, 0, 0, 0);      \
    __builtin_amdgcn_sched_barrier(0);                                                                           \
    asm volatile("s_cmp_lg_u32 %5, 0\n\ts_cbranch_scc0 1f\n\tbuffer_load_dwordx4 %0, %1, %2, %3 offen offset:%4 sc0 sc1\n1:" \
                 : "+v"(A[j]) : "v"(l16), "s"(rs), "s"(nbase + (j >> 2) * 4096), "i"((j & 3) * 1024), "s"(hv) : "memory", "scc"); \
    __builtin_amdgcn_sched_barrier(0);
    MS(0) MS(1) MS(2) MS(3) MS(4) MS(5) MS(6) MS(7)
    if (have) { WAITV(17); raise2(pend, 0); pend = 0; }
    if (owed) fresh = reduce(i - 1); else if (have) null_store(rs, l16);
    if (have) { pend = fresh; fresh = 0; }
    __builtin_amdgcn_sched_barrier(0);
    MS(8) MS(9) MS(10) MS(11) MS(12) MS(13) MS(14) MS(15)
#undef MS
    if (!good) { acc0 = __builtin_nanf(""); acc1 = acc0; }
    char* const pb = smem + (i & 1) * 16384;
    *(f32x4*)(pb + wslot + (wrot & 7) * 16) = f32x4{acc0[0], acc1[0], acc0[1], acc1[1]};
    *(f32x4*)(pb + wslot + ((wrot + 1) & 7) * 16) = f32x4{acc0[2], acc1[2], acc0[3], acc1[3]};
    asm volatile("s_waitcnt lgkmcnt(0)" ::: "memory");
    if (lane == 0) __hip_atomic_fetch_add(lcnt + (i & 1) * 32, 1u, __ATOMIC_RELAXED, __HIP_MEMORY_SCOPE_WORKGROUP);
    owed = have;
    if (!have) {
      if (more) {
        asm volatile("s_waitcnt vmcnt(0)" : "+v"(pf) :: "memory");
        raise2(pend, fresh);
        pend = fresh = 0;
        mask |= __ballot(all_ge(pf, call));
        if ((mask >> (i + 1)) & 1) { have = true; load_block(A, rs, l16, nbase); }
      }
      const unsigned f = reduce(i);
      if (have) {
        fresh = f;
      } else {
        if (f | pend | fresh) { WAITV(0); raise2(pend, fresh); raise2(f, 0); pend = fresh = 0; }
        if (more) {
          good = spin4<SL1>(rs, ws, foff, lane == i + 1, call, t0) && good;
          load_block(A, rs, l16, nbase);
          null_store(rs, l16);
        }
      }
    }
  }
  WAITV(0);
  STAMPE(3);
}

}

extern "C" void kernel_launch(int rank, int nranks, ncclComm_t comm, hipStream_t stream,
                              void* const* d_in, const int* in_sizes, int n_in,
                              void* d_out, int out_size, const int* in_dtypes, int out_dtype,
                              void* const* d_out_all, void* const* d_ws_all, size_t ws_bytes) {
  if (nranks != 8 || (unsigned)rank >= 8u || n_in < 2 || ws_bytes < WS_USED || out_size != 4096 * 512 ||
      in_sizes[0] != 512 * 4096 || in_sizes[1] != 4096 * 512) {
    fprintf(stderr, "unexpected problem shape\n");
    return;
  }
  constexpr int LDS_BYTES = 96 * 1024;
  HIP_CHECK(hipFuncSetAttribute((const void*)ag_gemm, hipFuncAttributeMaxDynamicSharedMemorySize, LDS_BYTES));
  Args a;
  a.x = (const float*)d_in[0];
  a.w = (const float*)d_in[1];
  for (int d = 0; d < 8; ++d) a.ws[d] = (char*)d_ws_all[rank ^ d];
  a.out = (char*)d_out;
  a.rank = rank;
  a.pad = 0;
  ag_gemm<<<256, 512, LDS_BYTES, stream>>>(a);
  HIP_CHECK(hipGetLastError());
}
```

```cpp
#include <hip/hip_runtime.h>
#include <rccl/rccl.h>
#include <cstdio>

#define HIP_CHECK(expr) do { hipError_t _e = (expr); if (_e != hipSuccess) { \
    fprintf(stderr, "HIP error %s at %s:%d\n", hipGetErrorString(_e), __FILE__, __LINE__); \
    return; } } while (0)

namespace {

typedef __attribute__((address_space(1))) unsigned gu32;
typedef __attribute__((address_space(3))) unsigned lu32;
typedef unsigned u32x4 __attribute__((ext_vector_type(4)));
typedef float f32x4 __attribute__((ext_vector_type(4)));
typedef float f32x2 __attribute__((ext_vector_type(2)));
typedef _Float16 h16x2 __attribute__((ext_vector_type(2)));
typedef _Float16 h16x8 __attribute__((ext_vector_type(8)));

constexpr int R1 = 23;
constexpr int R2 = 32 - R1;
constexpr int PE = 8;
constexpr int RAWJ = 12;
constexpr int WIN = 16;
constexpr int REPO = 6;
constexpr int REPF[4] = {6, 7, 8, 11};
constexpr int NOWN = 32 + 3 * R1 + 4 * R2;
constexpr int NFOR = 4 * R1 + 3 * R2;
constexpr int NJI = R1 * 8 - PE;
constexpr int JO = 64 + R2 * 8 - PE;
constexpr int NP = (NFOR - 3) / 2;
constexpr int JS = JO + NP, NJC = JS + NFOR - 2 * NP;
static_assert(NJC - JO <= 64 && NJC - JS <= 4 && NFOR <= 128 && NOWN <= 160 && (NOWN + REPO - 1) / REPO <= 52, "");
static_assert(4 * REPO + REPF[0] + REPF[1] + REPF[2] + REPF[3] == 56 && REPF[3] <= 16 && REPF[3] >= 8, "");
static_assert(PE <= 16 && WIN >= 10 && WIN <= 27 && NJI >= 64 && NJC <= 256 && NJI <= 256, "");

constexpr unsigned MiB = 1u << 20;
constexpr unsigned CNT_OFF = 0;
constexpr unsigned TMO_OFF = 16 << 10;
constexpr unsigned TICK_OFF = 20 << 10;
constexpr unsigned RF_OFF = 32 << 10;
constexpr unsigned SEAT_OFF = 24 << 10;
constexpr unsigned SG_OFF = 48 << 10;
constexpr unsigned XF_OFF = 256 << 10;
constexpr unsigned WF_OFF = 512 << 10;
constexpr unsigned PAR_STRIDE = 48 * MiB;
constexpr unsigned XS_OFF = 2 * MiB;
constexpr unsigned WS_OFF = XS_OFF + 32 * MiB;
constexpr unsigned PS_OFF = WS_OFF + 8 * MiB;
constexpr unsigned PSL = 256 + 14 * 1024;
constexpr unsigned OS_OFF = 98 * MiB;
constexpr unsigned WS_USED = 102 * MiB;
constexpr unsigned long long DEADLINE = 30000000ull;

struct Tables { unsigned pair[4][64]; unsigned quad[56]; unsigned char order[5][160]; };
constexpr int arrival(int d, int rb) {
  if (d == 0) return rb * 8 >= PE && rb * 8 + 8 <= PE + 64 ? -200 : -100;
  if (d < 4) return (rb + 1) * 8 - PE;
  return (rb - R1 + 1) * 8 <= PE ? 0 : 64 + (rb - R1 + 1) * 8 - PE;
}
constexpr bool wanted(int type, int d, int rb) {
  if (d == 0) return type == 0 || rb < R1;
  if (d < 4) return rb < R1;
  return rb >= R1 && (type == 0 || d != 3 + type);
}
constexpr Tables make_tables() {
  Tables t{};
  for (int type = 0; type < 5; ++type) {
    int n = 0;
    for (int rb = 0; rb < 32; ++rb)
      for (int d = 0; d < 8; ++d)
        if (wanted(type, d, rb)) {
          int e = d * 32 + rb, a = arrival(d, rb), i = n++;
          while (i > 0 && arrival(t.order[type][i - 1] >> 5, t.order[type][i - 1] & 31) > a) {
            t.order[type][i] = t.order[type][i - 1];
            --i;
          }
          t.order[type][i] = (unsigned char)e;
        }
  }
  {
    int used[5] = {0, 0, 0, 0, 0}, nf = 0, i = 0;
    const int tot[5] = {REPO, REPF[0], REPF[1], REPF[2], REPF[3] - 8};
    auto put = [&](unsigned v) { if (i % 7 == 0) { t.quad[i] = 7 | (REPF[3] - 8 + i / 7) << 8 | 255 << 16; ++i; } t.quad[i++] = v; };
    while (i < 56) {
      int best = -1;
      for (int c = 0; c < 5; ++c)
        if (used[c] < tot[c] && (best < 0 || (2 * used[c] + 1) * tot[best] < (2 * used[best] + 1) * tot[c])) best = c;
      if (best == 0) for (int cb = 0; cb < 4; ++cb) put(cb | used[0] << 8 | 255 << 16);
      else put((3 + best) | used[best] << 8 | nf++ << 16);
      ++used[best];
    }
  }
  for (int q = 0; q < 4; ++q)
    for (int m = 0; m < 64; ++m) t.pair[q][m] = m < NP ? t.order[1 + q][2 * m] | (unsigned)t.order[1 + q][2 * m + 1] << 8 : t.order[1 + q][NP + m];
  return t;
}
__constant__ const Tables TB = make_tables();

struct Args {
  const float* x;
  const float* w;
  char* ws[8];
  char* out;
  int rank;
  unsigned links;
};

#define WAITV(n) asm volatile("s_waitcnt vmcnt(%0)" ::"i"(n) : "memory")
#define CBAR() asm volatile("" ::: "memory")
constexpr int SYS = 17;
constexpr unsigned OOB = 0xF0000000u;

__device__ __forceinline__ unsigned pk(float a, float b) {
  h16x2 h = {(_Float16)a, (_Float16)b};
  return __builtin_bit_cast(unsigned, h);
}
__device__ __forceinline__ u32x4 pk8(f32x4 a, f32x4 b) {
  return u32x4{pk(a[0], a[1]), pk(a[2], a[3]), pk(b[0], b[1]), pk(b[2], b[3])};
}
__device__ __forceinline__ void st16(char* base, unsigned voff, u32x4 v) {
  asm volatile("global_store_dwordx4 %0, %1, %2 sc0 sc1\n\ts_nop 1" ::"v"(voff), "v"(v), "s"(base) : "memory");
}
template <int IMM>
__device__ __forceinline__ void st32(char* base, unsigned voff, u32x4 v, u32x4 w) {
  asm volatile("global_store_dwordx4 %0, %1, %3 offset:%4 sc0 sc1\n\tglobal_store_dwordx4 %0, %2, %3 offset:%5 sc0 sc1\n\ts_nop 1"
               ::"v"(voff), "v"(v), "v"(w), "s"(base), "i"(IMM), "i"(IMM + 1024) : "memory");
}
__device__ __forceinline__ void put_flag(char* p, unsigned v) {
  __hip_atomic_store((gu32*)p, v, __ATOMIC_RELAXED, __HIP_MEMORY_SCOPE_SYSTEM);
}
__device__ __forceinline__ bool all_ge(u32x4 f, unsigned call) {
  return f[0] >= call && f[1] >= call && f[2] >= call && f[3] >= call;
}
__device__ __forceinline__ unsigned long long now() { return __builtin_amdgcn_s_memrealtime(); }
__device__ __forceinline__ bool expired(char* ws, unsigned call, unsigned long long t0) {
  gu32* tmo = (gu32*)(ws + TMO_OFF);
  if (__builtin_amdgcn_readfirstlane(__hip_atomic_load(tmo, __ATOMIC_RELAXED, __HIP_MEMORY_SCOPE_AGENT)) != call &&
      (unsigned)(now() - t0) <= (unsigned)DEADLINE) return false;
  __hip_atomic_store(tmo, call, __ATOMIC_RELAXED, __HIP_MEMORY_SCOPE_AGENT);
  return true;
}
template <int SLEEP>
__device__ __forceinline__ bool spin4(__amdgpu_buffer_rsrc_t rs, char* ws, unsigned off, bool need, unsigned call,
                                      unsigned long long t0) {
  for (unsigned it = 0;; ++it) {
    bool ok = true;
    CBAR();
    if (need) ok = all_ge(__builtin_amdgcn_raw_buffer_load_b128(rs, off, 0, SYS), call);
    if (__all(ok)) { CBAR(); return true; }
    if ((it & 63) == 63 && expired(ws, call, t0)) return false;
    __builtin_amdgcn_s_sleep(SLEEP);
  }
}

constexpr unsigned HOLE = 0x60606060u;
__device__ __forceinline__ void swap_bits(unsigned& x, unsigned& y, unsigned m) {
  const unsigned nx = (m & y) | (~m & x), ny = (m & x) | (~m & y);
  x = nx; y = ny;
}
__device__ __forceinline__ unsigned rotr(unsigned x, unsigned n) { return __builtin_amdgcn_alignbit(x, x, n); }

__device__ __forceinline__ unsigned pack_slice(const u32x4 (&f)[16], char* ws, unsigned off, char* list, int lane) {
  unsigned sum = 0;
#pragma unroll
  for (int u = 0; u < 8; ++u) sum = __builtin_amdgcn_sad_u8(__builtin_amdgcn_perm(f[2 * u + 1][u & 3], f[2 * u][u & 3], 0x07050301u) & 0x7C7C7C7Cu, 0, sum);
  sum += __builtin_amdgcn_update_dpp(0u, sum, 0x111, 0xf, 0xf, false);
  sum += __builtin_amdgcn_update_dpp(0u, sum, 0x112, 0xf, 0xf, false);
  sum += __builtin_amdgcn_update_dpp(0u, sum, 0x114, 0xf, 0xf, false);
  sum += __builtin_amdgcn_update_dpp(0u, sum, 0x118, 0xf, 0xf, false);
  const int tot = __builtin_amdgcn_readlane(sum, 15) + __builtin_amdgcn_readlane(sum, 31) + __builtin_amdgcn_readlane(sum, 47) + __builtin_amdgcn_readlane(sum, 63);
  const int e0 = (tot - 42189 + 8191) >> 13;
  const unsigned base = e0 < 0 ? 0 : e0 > 24 ? 24 : e0;
  *(unsigned*)(list + lane * 4) = 0;
  unsigned P[8];
#pragma unroll
  for (int h = 0; h < 2; ++h) {
    const unsigned o = off + 256 + h * 7168 + lane * 16;
    u32x4 HH[3];
#pragma unroll
    for (int v = 0; v < 4; ++v) {
      const int u = h * 4 + v;
      u32x4 L;
      unsigned H[4];
#pragma unroll
      for (int j = 0; j < 4; ++j) {
        L[j] = __builtin_amdgcn_perm(f[2 * u + 1][j], f[2 * u][j], 0x06040200u);
        H[j] = __builtin_amdgcn_perm(f[2 * u + 1][j], f[2 * u][j], 0x07050301u) - base * 0x04040404u;
      }
      unsigned g = H[3];
#pragma unroll
      for (int j = 0; j < 3; ++j) { g = rotr(g, 30); swap_bits(H[j], g, HOLE); HH[j][v] = H[j]; }
      st16(ws, o + (3 + v) * 1024, L);
      P[u] = g;
    }
#pragma unroll
    for (int j = 0; j < 3; ++j) st16(ws, o + j * 1024, HH[j]);
  }
  unsigned lo[4], hi[4], cnt = 0;
#pragma unroll
  for (int v = 0; v < 4; ++v) {
    lo[v] = (0x55555555u & rotr(P[2 * v], 1)) | (0xAAAAAAAAu & P[2 * v + 1]);
    hi[v] = (0x55555555u & rotr(P[2 * v], 2)) | (0xAAAAAAAAu & rotr(P[2 * v + 1], 1));
    cnt += __builtin_popcount(lo[v] | hi[v]);
  }
  unsigned pos = cnt;
  pos += __builtin_amdgcn_update_dpp(0u, pos, 0x111, 0xf, 0xf, false);
  pos += __builtin_amdgcn_update_dpp(0u, pos, 0x112, 0xf, 0xf, false);
  pos += __builtin_amdgcn_update_dpp(0u, pos, 0x114, 0xf, 0xf, false);
  pos += __builtin_amdgcn_update_dpp(0u, pos, 0x118, 0xf, 0xf, false);
  pos += __builtin_amdgcn_update_dpp(0u, pos, 0x142, 0xa, 0xf, false);
  pos += __builtin_amdgcn_update_dpp(0u, pos, 0x143, 0xc, 0xf, false);
  if ((unsigned)__builtin_amdgcn_readlane(pos, 63) > 128) return 32;
  char* q = list + (pos - cnt) * 2;
#pragma unroll
  for (int v = 0; v < 4; ++v)
    for (unsigned m = lo[v] | hi[v]; m; m &= m - 1, q += 2) {
      const unsigned s = __builtin_ctz(m);
      *(unsigned short*)q = (unsigned short)((lane * 4 + v) << 7 | s << 2 | ((hi[v] >> s) & 1) << 1 | ((lo[v] >> s) & 1));
    }
  CBAR();
  if (lane < 16) st16(ws, off + lane * 16, *(const u32x4*)(list + lane * 16));
  return base;
}

static_assert(WS_OFF == XS_OFF + 2048 * 16384 && WF_OFF == XF_OFF + 2048 * 128, "");
__device__ __forceinline__ unsigned landing(unsigned j, unsigned d) {
  return j + (d < 4 ? d * 256 + PE : j < 64 ? 2048 + d * 64 : d * 256 + R1 * 8 + PE - 64);
}
__device__ __forceinline__ unsigned strip_dst(unsigned e, unsigned d, int rank) {
  return ((rank ^ d ^ ((e >> 5) & 7)) * 512 + (e & 31) * 16) * 2048 + (d - 4) * 512;
}

constexpr int NSLOT = 5;
constexpr unsigned PLANE_LDS = NSLOT * 16384;
constexpr unsigned CTL_LDS = PLANE_LDS + 6144;

template <int IMM>
__device__ __forceinline__ void dma(unsigned lds, unsigned voff, const char* base) {
  unsigned keep;
  asm volatile("s_mov_b32 %0, m0\n\ts_mov_b32 m0, %1\n\ts_waitcnt vmcnt(%5)\n\tglobal_load_lds_dwordx4 %2, %3 offset:%4 sc0 sc1\n\ts_mov_b32 m0, %0"
               : "=&s"(keep) : "s"(lds), "v"(voff), "s"(base), "i"(IMM), "i"(WIN - 1) : "memory");
}
__device__ __forceinline__ void dma_masked(unsigned lds, unsigned voff, const char* base, unsigned long long lanes) {
  unsigned keep;
  asm volatile("s_mov_b32 %0, m0\n\ts_mov_b32 m0, %1\n\ts_mov_b64 exec, %4\n\ts_waitcnt vmcnt(%5)\n\t"
               "global_load_lds_dwordx4 %2, %3 sc0 sc1\n\ts_mov_b64 exec, -1\n\ts_mov_b32 m0, %0"
               : "=&s"(keep) : "s"(lds), "v"(voff), "s"(base), "s"(lanes), "i"(WIN - 1) : "memory");
}

__device__ __forceinline__ void loader(const Args& a, char* smem, unsigned d, unsigned w, int lane, unsigned call,
                                       unsigned long long t0) {
  char* const ws = a.ws[0];
  const char* const peer = d == 1 ? a.ws[1] : d == 2 ? a.ws[2] : d == 3 ? a.ws[3] : d == 4 ? a.ws[4] : d == 5 ? a.ws[5] : d == 6 ? a.ws[6] : a.ws[7];
  const unsigned l16 = lane * 16, par = (call & 1) * PAR_STRIDE, nj = d < 4 ? NJI : JS, jo = d < 4 ? 256 : JO, jw = d < 4 ? 0 : 64;
  const unsigned rf = RF_OFF + ((call & 1) * 8 + d) * 1024, tko = TICK_OFF + d * 128;
  const unsigned lds0 = (unsigned)(size_t)(lu32*)smem;
  volatile lu32* const ctl = (volatile lu32*)(smem + CTL_LDS);
  const unsigned xs = d < 4 ? PE : R1 * 8 + PE - 64;
  const unsigned rawX = par + XS_OFF + xs * 16384, rawW = par + WS_OFF + (d - 4) * 64 * 16384, rawO = OS_OFF + ((d - 4) * 64 - JO) * 16384;
  const unsigned pakX = par + PS_OFF + xs * PSL, pakW = par + PS_OFF + d * 64 * PSL;
  const unsigned thrS = call << 6, thrO = 8 * ((call + 1) >> 1);
  auto words = [&](unsigned j) {
    const unsigned k = ((j & 240) + (lane & 31)) & 255;
    return (k >= jo && k < nj ? peer : ws) + rf + k * 4;
  };
  __builtin_amdgcn_s_setprio(3);
  int landed = 0;
  auto publish = [&](int f) { if (f > landed) { landed = f; ctl[0] = f; } };
  const unsigned tbase = (call - 1) * nj - 4;
  unsigned j = w, tk = 0, got = 0, info = 0, freed = 0;
  int n = 0, slot = 0, lap = 0;
  for (;;) {
    if (j >= nj || info < (j >= jo ? thrO : thrS)) {
      unsigned fresh = 0;
      asm volatile("global_load_dword %0, %1, off sc0 sc1\n\ts_waitcnt vmcnt(16)" : "+v"(fresh) : "v"(words(j)) : "memory");
      publish(3 * n - 2);
      WAITV(9); publish(3 * n - 1);
      WAITV(1); publish(3 * n);
      asm volatile("s_waitcnt vmcnt(0)" : "+v"(fresh) :: "memory");
      if (j >= nj) break;
      info = __builtin_amdgcn_readlane(fresh, j & 15);
      bool ok = true;
      for (unsigned it = 0; info < (j >= jo ? thrO : thrS); ++it) {
        if ((it & 63) == 63 && expired(ws, call, t0)) { ok = false; break; }
        info = __builtin_amdgcn_readfirstlane(__hip_atomic_load((const gu32*)((j >= jo ? peer : ws) + rf + j * 4), __ATOMIC_RELAXED, __HIP_MEMORY_SCOPE_SYSTEM));
      }
      CBAR();
      if (!ok) {
        if (j >= jo) put_flag(a.out + strip_dst(TB.pair[d & 3][(j - JO) & 63], d, a.rank), 0x7fc00000u);
        break;
      }
      if (n == 0) while ((unsigned)(now() - t0) < 250) __builtin_amdgcn_s_sleep(1);
    }
    if (j >= jo) info = 32;
    const bool raw = info & 32;
    const unsigned src = (raw ? (j >= jo ? rawO : j < jw ? rawW : rawX) : j < jw ? pakW : pakX) + j * (raw ? 16384 : PSL);
    while ((int)__builtin_amdgcn_readfirstlane(freed) < 2 * lap) freed = ctl[4 + slot];
    *(volatile __attribute__((address_space(3))) unsigned long long*)(ctl + 16 + 2 * slot) = (unsigned long long)info << 32 | j;
    const unsigned lds = lds0 + slot * 16384;
    const unsigned v0 = src + l16 - (raw ? 0 : 768), v1 = v0 + (raw ? 8192 : 7168);
    const char* const b8 = raw ? peer : ws;
#define OP(i) if (i == (2 + WIN) % 18) { WAITV(WIN - 1); publish(3 * (n - (2 + WIN) / 18) + 1); } \
              if (i == (9 + WIN) % 18) { WAITV(WIN - 1); publish(3 * (n - (9 + WIN) / 18) + 2); } \
              if (i == (17 + WIN) % 18) { WAITV(WIN - 1); publish(3 * (n - (17 + WIN) / 18) + 3); }
    tk = 1;
    OP(0) asm volatile("s_mov_b64 exec, 1\n\ts_nop 3\n\ts_waitcnt vmcnt(%3)\n\tglobal_atomic_add %0, %1, %0, %2 sc0\n\ts_mov_b64 exec, -1" : "+v"(tk) : "v"(tko), "s"(ws), "i"(WIN - 1), "s"(peer), "s"(b8) : "memory");
    OP(1) asm volatile("s_waitcnt vmcnt(%2)\n\tglobal_load_dword %0, %1, off sc0 sc1" : "+v"(got) : "v"(words(j)), "i"(WIN - 1) : "memory");
    OP(2) dma_masked(lds, src + l16, peer, raw ? ~0ull : 0xFFFFull);
    OP(3) dma<1024>(lds, v0, peer);
    OP(4) dma<2048>(lds, v0, peer);
    OP(5) dma<3072>(lds, v0, peer);
    OP(6) dma<0>(lds + 4096, v0 + 4096, peer);
    OP(7) dma<1024>(lds + 4096, v0 + 4096, peer);
    OP(8) dma<2048>(lds + 4096, v0 + 4096, peer);
    OP(9) dma<3072>(lds + 4096, v0 + 4096, peer);
    OP(10) dma_masked(lds + 8192, raw ? v1 : CNT_OFF, b8, raw ? ~0ull : 1ull);
    OP(11) dma<1024>(lds + 8192, v1, peer);
    OP(12) dma<2048>(lds + 8192, v1, peer);
    OP(13) dma<3072>(lds + 8192, v1, peer);
    OP(14) dma<0>(lds + 12288, v1 + 4096, peer);
    OP(15) dma<1024>(lds + 12288, v1 + 4096, peer);
    OP(16) dma<2048>(lds + 12288, v1 + 4096, peer);
    OP(17) dma<3072>(lds + 12288, v1 + 4096, peer);
#undef OP
    ++n;
    if (++slot == NSLOT) { slot = 0; ++lap; }
    freed = ctl[4 + slot];
    asm volatile("s_waitcnt vmcnt(16)" : "+v"(got), "+v"(tk) :: "memory");
    publish(3 * n - 3);
    const unsigned jc = j;
    j = __builtin_amdgcn_readfirstlane(tk) - tbase;
    info = j - (jc & 240) < 32 ? __builtin_amdgcn_readlane(got, (j - (jc & 240)) & 31) : 0;
  }
  ctl[1] = 1;
}

__device__ __forceinline__ void unpacker(const Args& a, char* smem, unsigned d, int lane, int wave, unsigned call) {
  char* const ws = a.ws[0];
  const unsigned h = (wave - 1) & 1, l16 = lane * 16, lout = (lane >> 5) * 2048 + (lane & 31) * 16;
  volatile lu32* const ctl = (volatile lu32*)(smem + CTL_LDS);
  char* const plane = smem + PLANE_LDS + (wave - 1) * 1024;
  const u32x4 zero = {0, 0, 0, 0};
  *(u32x4*)(plane + l16) = zero;
  const unsigned odst = strip_dst(TB.pair[d & 3][lane] >> (8 * h), d, a.rank);
  auto await = [&](unsigned thirds) -> bool {
    for (;;) {
      const unsigned all = __builtin_amdgcn_readfirstlane(ctl[1]), landed = __builtin_amdgcn_readfirstlane(ctl[0]);
      if (landed >= thirds) { __builtin_amdgcn_s_setprio(2); CBAR(); return true; }
      __builtin_amdgcn_s_setprio(0);
      if (all) return false;
      __builtin_amdgcn_s_sleep(1);
    }
  };
  for (unsigned n = (wave - 1) >> 1; await(3 * n + 1); n += 3) {
    const unsigned slot = n % NSLOT;
    const unsigned long long job = *(const unsigned long long*)(smem + CTL_LDS + 64 + slot * 8);
    const unsigned lst = *(const unsigned*)(smem + slot * 16384 + lane * 4);
    const unsigned j = __builtin_amdgcn_readfirstlane((unsigned)job), info = __builtin_amdgcn_readfirstlane((unsigned)(job >> 32));
    const bool strip = d >= 4 && j >= JO;
    const unsigned flag = strip ? 0 : XF_OFF + landing(j, d) * 128;
    const unsigned dst = strip ? __builtin_amdgcn_readlane(odst, (j - JO) & 63) : (call & 1) * PAR_STRIDE + XS_OFF + landing(j, d) * 16384 + h * 8192;
    const char* const p = smem + slot * 16384 + h * 8192 + l16;
    if (info & 32) {
      char* const db = strip ? a.out : ws;
      const unsigned o = dst + (strip ? lout : l16), step = strip ? 4096 : 1024;
      asm volatile("s_nop 4" :: "s"(db));
      if (!await(3 * n + 2 + h)) break;
      u32x4 r[8];
#pragma unroll
      for (int i = 0; i < 8; ++i) r[i] = *(const u32x4*)(p + i * 1024);
#pragma unroll
      for (int i = 0; i < 8; ++i) st16(db, o + i * step, r[i]);
    } else {
#pragma unroll
      for (int c = 0; c < 2; ++c) {
        const unsigned e = c ? lst >> 16 : lst & 0xffff;
        if (((e >> 8) & 1) == h)
          __hip_atomic_fetch_or((lu32*)(plane + ((e >> 9) * 16 | ((e >> 4) & 8) | (e & 4))), rotr((e & 3) << ((e >> 2) & 30), 31),
                                __ATOMIC_RELAXED, __HIP_MEMORY_SCOPE_WORKGROUP);
      }
      CBAR();
      const u32x4 P = *(const u32x4*)(plane + l16);
      CBAR();
      *(u32x4*)(plane + l16) = zero;
      const unsigned base = (info & 31) * 0x04040404u, o = dst + l16 + 4096;
      if (!await(3 * n + 2 + h)) break;
      u32x4 r[8];
#pragma unroll
      for (int i = 1; i < 8; ++i) r[i] = *(const u32x4*)(p + i * 1024);
#define UNIT(v) {                                                                                                          \
        unsigned h0 = r[1][v], h1 = r[2][v], h2 = r[3][v], g = P[v];                                                       \
        swap_bits(h2, g, HOLE); g = rotr(g, 2); swap_bits(h1, g, HOLE); g = rotr(g, 2); swap_bits(h0, g, HOLE); g = rotr(g, 2); \
        h0 += base; h1 += base; h2 += base; g += base;                                                                     \
        const u32x4 L = r[4 + v];                                                                                          \
        st32<v * 2048 - 4096>(ws, o, u32x4{__builtin_amdgcn_perm(h0, L[0], 0x05010400u), __builtin_amdgcn_perm(h1, L[1], 0x05010400u), \
                                           __builtin_amdgcn_perm(h2, L[2], 0x05010400u), __builtin_amdgcn_perm(g, L[3], 0x05010400u)}, \
                                     u32x4{__builtin_amdgcn_perm(h0, L[0], 0x07030602u), __builtin_amdgcn_perm(h1, L[1], 0x07030602u), \
                                           __builtin_amdgcn_perm(h2, L[2], 0x07030602u), __builtin_amdgcn_perm(g, L[3], 0x07030602u)}); }
      UNIT(0) UNIT(1) UNIT(2) UNIT(3)
#undef UNIT
    }
    if (lane == 0) __hip_atomic_fetch_add((lu32*)(ctl + 4 + slot), 1u, __ATOMIC_RELAXED, __HIP_MEMORY_SCOPE_WORKGROUP);
    if (flag) {
      WAITV(0);
      if (lane < 2) put_flag(ws + flag + h * 8 + lane * 4, call);
    }
  }
}

__device__ __forceinline__ void closer(const Args& a, unsigned d, int lane, int wave, unsigned call, unsigned long long t0) {
  char* const ws = a.ws[0];
  const char* const peer = d == 4 ? a.ws[4] : d == 5 ? a.ws[5] : d == 6 ? a.ws[6] : a.ws[7];
  const unsigned k = wave >> 1, h = wave & 1;
  const unsigned o = strip_dst(TB.pair[d - 4][NP + k], d, a.rank) + h * 16384 + (lane >> 5) * 2048 + (lane & 31) * 16;
  const unsigned src = OS_OFF + ((d - 4) * 128 + 2 * NP + k) * 8192 + h * 4096 + lane * 16;
  const gu32* const word = (const gu32*)(peer + SG_OFF + ((d - 4) * 8 + k) * 128) + (lane & 3);
  for (unsigned it = 0; __builtin_amdgcn_readfirstlane(__hip_atomic_load((gu32*)(ws + TICK_OFF + d * 128), __ATOMIC_RELAXED, __HIP_MEMORY_SCOPE_AGENT)) < call * JS - 12; ++it) {
    if ((it & 15) == 15 && expired(ws, call, t0)) break;
    __builtin_amdgcn_s_sleep(32);
  }
  for (unsigned it = 0;; ++it) {
    unsigned v = call;
    if (lane < 4) v = __hip_atomic_load(word, __ATOMIC_RELAXED, __HIP_MEMORY_SCOPE_SYSTEM);
    if (__all(v >= call)) break;
    if ((it & 63) == 63 && expired(ws, call, t0)) {
      put_flag(a.out + o, 0x7fc00000u);
      return;
    }
  }
  u32x4 r[4];
  asm volatile("global_load_dwordx4 %0, %4, %5 sc0 sc1\n\tglobal_load_dwordx4 %1, %4, %5 offset:1024 sc0 sc1\n\t"
               "global_load_dwordx4 %2, %4, %5 offset:2048 sc0 sc1\n\tglobal_load_dwordx4 %3, %4, %5 offset:3072 sc0 sc1\n\ts_waitcnt vmcnt(0)"
               : "=&v"(r[0]), "=&v"(r[1]), "=&v"(r[2]), "=&v"(r[3]) : "v"(src), "s"(peer) : "memory");
#pragma unroll
  for (int i = 0; i < 4; ++i) st16(a.out, o + i * 4096, r[i]);
}

template <int N>
__device__ __forceinline__ void load_x(f32x4 (&s)[2 * N], const float* x, int rb, int k0, int lane) {
  const f32x4* p = (const f32x4*)(x + (size_t)(rb * 16 + (lane & 15)) * 4096 + k0 + (lane >> 4) * 4);
#pragma unroll
  for (int f = 0; f < N; ++f) { s[2 * f] = p[8 * f]; s[2 * f + 1] = p[8 * f + 4]; }
}
template <int IMM>
__device__ __forceinline__ void ldbuf(u32x4& v, __amdgpu_buffer_rsrc_t rs, unsigned voff, unsigned soff) {
  asm volatile("buffer_load_dwordx4 %0, %1, %2, %3 offen offset:%4 sc0 sc1" : "+v"(v) : "v"(voff), "s"(rs), "s"(soff), "i"(IMM) : "memory");
}
__device__ __forceinline__ void load_block(u32x4 (&A)[16], __amdgpu_buffer_rsrc_t rs, unsigned voff, unsigned soff) {
  asm volatile("s_nop 4");
#define L4(j) ldbuf<0>(A[j], rs, voff, soff + j * 1024); ldbuf<1024>(A[j + 1], rs, voff, soff + j * 1024); \
              ldbuf<2048>(A[j + 2], rs, voff, soff + j * 1024); ldbuf<3072>(A[j + 3], rs, voff, soff + j * 1024);
  L4(0) L4(4) L4(8) L4(12)
#undef L4
}
__device__ __forceinline__ void null_store(__amdgpu_buffer_rsrc_t rs, unsigned v) {
  asm volatile("buffer_store_dword %0, %0, %1, %2 offen" :: "v"(v), "s"(rs), "s"(OOB) : "memory");
}
__device__ __forceinline__ unsigned next_call(gu32* cnt) {
  const unsigned call = __builtin_amdgcn_readfirstlane(__hip_atomic_load(cnt, __ATOMIC_RELAXED, __HIP_MEMORY_SCOPE_AGENT)) + 1;
  __hip_atomic_store(cnt, call, __ATOMIC_RELAXED, __HIP_MEMORY_SCOPE_AGENT);
  return call;
}

__global__ void __launch_bounds__(512, 2) ag_gemm(const Args a) {
  extern __shared__ __attribute__((aligned(16))) char smem[];
  const int lane = threadIdx.x & 63;
  const int wave = __builtin_amdgcn_readfirstlane(threadIdx.x >> 6);
  const int xcd = blockIdx.x & 7, slot = blockIdx.x >> 3;
  char* const ws = a.ws[0];
  const unsigned long long t0 = now();
  gu32* const cnt = (gu32*)(ws + CNT_OFF) + blockIdx.x * 8 + wave;
  const unsigned l16 = lane * 16;
  const __amdgpu_buffer_rsrc_t rs = __builtin_amdgcn_make_buffer_rsrc(ws, 0, WS_USED, 0x00020000);
  auto lane_peer = [&] {
    unsigned lo = 0, hi = 0;
#pragma unroll
    for (int d = 1; d < 8; ++d) {
      unsigned l = (unsigned)(size_t)a.ws[d], h = (unsigned)((size_t)a.ws[d] >> 32);
      asm volatile("" : "+s"(l), "+s"(h));
      lo = lane == d ? l : lo;
      hi = lane == d ? h : hi;
    }
    return (char*)((size_t)hi << 32 | lo);
  };

  constexpr int NOP = PE * 4, NU = 5 * RAWJ, NC = 224 - NOP;
  static_assert(NU <= 96, "");
  unsigned call;
  if (blockIdx.x < NOP) {
    const bool on = wave < 4;
    f32x4 s[2], s2[2];
    if (on) {
      load_x<1>(s, a.x, blockIdx.x >> 5, (blockIdx.x & 31) * 128 + wave * 32, lane);
      load_x<1>(s2, a.x, R1 + (blockIdx.x >> 5), (blockIdx.x & 31) * 128 + wave * 32, lane);
    }
    call = next_call(cnt);
    if (on) {
      const unsigned o1 = XS_OFF + (call & 1) * PAR_STRIDE + blockIdx.x * 4096 + wave * 1024 + l16, o2 = o1 + R1 * 131072;
      const u32x4 h1 = pk8(s[0], s[1]), h2 = pk8(s2[0], s2[1]);
      for (int i = 0; i < wave; ++i) __builtin_amdgcn_s_sleep(18);
      st16(a.ws[4] + 16 * MiB, o2, h2); st16(a.ws[1] + 4 * MiB, o1, h1);
      st16(a.ws[5] + 20 * MiB, o2, h2); st16(a.ws[2] + 8 * MiB, o1, h1);
      st16(a.ws[6] + 24 * MiB, o2, h2); st16(a.ws[3] + 12 * MiB, o1, h1);
      st16(a.ws[7] + 28 * MiB, o2, h2);
    }
    char* const lp = lane_peer();
    asm volatile("s_waitcnt vmcnt(0)\n\ts_barrier" ::: "memory");
    if (on && wave == 0 && lane >= 1 && lane < 8)
      put_flag(lp + XF_OFF + (lane * 256 + (lane >= 4 ? R1 * 8 : 0) + (blockIdx.x >> 2)) * 128 + (blockIdx.x & 3) * 4, call);
  } else if (blockIdx.x < NOP + NU) {
    const int k = (blockIdx.x - NOP) / 5, i = blockIdx.x - NOP - k * 5;
    u32x4 f0, f1;
    if (i == 0) {
      f32x4 s[4];
      load_x<2>(s, a.x, (PE + k) >> 3, ((PE + k) & 7) * 512 + wave * 64, lane);
      call = next_call(cnt);
      f0 = pk8(s[0], s[1]); f1 = pk8(s[2], s[3]);
    } else {
      const f32x2* p = (const f32x2*)(a.w + (size_t)((k & 15) * 256 + wave * 32 + (lane >> 4) * 4) * 512 + ((i - 1) * 4 + (k >> 4)) * 32 + (lane & 15) * 2);
      f32x2 s[8];
#pragma unroll
      for (int j = 0; j < 8; ++j) s[j] = p[((j & 3) + (j >> 2) * 16) * 256];
      call = next_call(cnt);
      f0 = u32x4{pk(s[0][0], s[1][0]), pk(s[2][0], s[3][0]), pk(s[4][0], s[5][0]), pk(s[6][0], s[7][0])};
      f1 = u32x4{pk(s[0][1], s[1][1]), pk(s[2][1], s[3][1]), pk(s[4][1], s[5][1]), pk(s[6][1], s[7][1])};
    }
    const unsigned gs = i ? 2048 + (i - 1) * 64 + k : PE + k;
    const unsigned o = XS_OFF + (call & 1) * PAR_STRIDE + gs * 16384 + wave * 2048 + l16;
    st16(ws, o, f0); st16(ws, o + 1024, f1);
    char* const lp = lane_peer();
    asm volatile("s_waitcnt vmcnt(0)\n\ts_barrier" ::: "memory");
    if (wave == 7) {
      if (i ? lane == 3 + i : lane >= 1 && lane < 4) put_flag(lp + RF_OFF + ((call & 1) * 8 + lane) * 1024 + k * 4, call << 6 | 32);
      if (lane < 4) put_flag(ws + XF_OFF + gs * 128 + lane * 4, call);
    }
  } else {
    call = next_call(cnt);
  }

  if (slot >= 28) {
    if (threadIdx.x < 16) {
      unsigned seat;
      asm volatile("s_getreg_b32 %0, hwreg(HW_REG_XCC_ID, 0, 3)" : "=s"(seat));
      seat += (blockIdx.x - 224) & 24;
      if (lane == 15)
#pragma clang loop unroll(disable)
        for (int i = 0; i < 32 && __hip_atomic_exchange((gu32*)(ws + SEAT_OFF + seat * 128), call, __ATOMIC_RELAXED, __HIP_MEMORY_SCOPE_AGENT) == call; ++i)
          seat = (seat + 1) & 31;
      *(unsigned*)(smem + CTL_LDS + lane * 4) = lane == 15 ? seat : 0;
    }
    asm volatile("s_waitcnt lgkmcnt(0)\n\ts_barrier" ::: "memory");
    const unsigned seat = __builtin_amdgcn_readfirstlane(*(const unsigned*)(smem + CTL_LDS + 60));
    const unsigned d = a.links ? (a.links >> (4 * ((seat >> 3) + (seat & 4)))) & 7 : ((seat >> 3) * 2 + ((seat >> 1) & 1) + 1) & 7;
    const unsigned w = a.links ? seat & 3 : ((seat >> 1) & 2) + (seat & 1);
    if (d) {
      if (wave == 0) loader(a, smem, d, w, lane, call, t0);
      else if (wave < 7) unpacker(a, smem, d, lane, wave, call);
    } else if (wave < 2 * (NJC - JS)) {
      closer(a, 4 + w, lane, wave, call, t0);
    }
    return;
  }

  const int id = xcd * 28 + slot;
  const unsigned role = TB.quad[id >> 2];
  const bool isown = (role & 255) < 4;
  const int rep = (role >> 8) & 255, g = isown ? id & 3 : (role & 255) - 4;
  const int type = isown ? 0 : 1 + (id & 3), cg = isown ? (role & 255) * 4 + g : 12 + type * 4 + g;
  {
    constexpr int NF = 192 - 16 * REPO;
    const int t = NU + (wave >= 2 ? 2 * NC + (wave - 2) * NF + (role >> 16) * 4 + (id & 3) : blockIdx.x < NOP ? 512 : wave * NC + 223 - (int)blockIdx.x);
    if (t < 512) {
      int sl;
      if (t < 320) { const int k = t / 5, i = t - k * 5; sl = i ? 256 + (i - 1) * 64 + k : PE + k; }
      else if (t < 320 + 2 * PE) sl = t - 320 < PE ? t - 320 : R1 * 8 + t - 320 - PE;
      else { sl = t - 2 * PE - 320 + PE + 64; if (sl >= R1 * 8) sl += PE; }
      while (now() - t0 < 30 + ((unsigned)((t - NU) * 800) >> 9)) __builtin_amdgcn_s_sleep(2);
      u32x4 f[16];
      if (sl < 256) {
        f32x4 s[32];
        load_x<16>(s, a.x, sl >> 3, (sl & 7) * 512, lane);
#pragma unroll
        for (int i = 0; i < 16; ++i) f[i] = pk8(s[2 * i], s[2 * i + 1]);
      } else {
        const f32x2* p = (const f32x2*)(a.w + (size_t)((sl & 15) * 256 + (lane >> 4) * 4) * 512 + ((sl - 256) >> 4) * 32 + (lane & 15) * 2);
        f32x2 s[64];
#pragma unroll
        for (int i = 0; i < 64; ++i) s[i] = p[((i >> 3) * 32 + (i & 3) + ((i >> 2) & 1) * 16) * 256];
#pragma unroll
        for (int i = 0; i < 16; ++i) {
          const int b = (i >> 1) * 8, c = i & 1;
          f[i] = u32x4{pk(s[b][c], s[b + 1][c]), pk(s[b + 2][c], s[b + 3][c]), pk(s[b + 4][c], s[b + 5][c]), pk(s[b + 6][c], s[b + 7][c])};
        }
      }
      const unsigned par = (call & 1) * PAR_STRIDE, gs = sl < 256 ? sl : 2048 - 256 + sl;
#pragma unroll
      for (int i = 0; i < 16; ++i) st16(ws, par + XS_OFF + gs * 16384 + l16 + i * 1024, f[i]);
      const int job = sl >= 256 ? sl & 63 : sl < R1 * 8 ? sl - PE : sl - R1 * 8 < PE ? -1 : sl - R1 * 8 - PE + 64;
      const bool mine = sl >= 256 ? lane == sl >> 6 : sl < R1 * 8 ? lane >= 1 && lane < 4 : lane >= 4 && lane < 8;
      const unsigned info = job < 0 ? 32 : pack_slice(f, ws, par + PS_OFF + sl * PSL, smem + 36864 + wave * 256, lane);
      char* const lp = lane_peer();
      WAITV(0);
      if (job >= 0 && mine) put_flag(lp + RF_OFF + ((call & 1) * 8 + lane) * 1024 + job * 4, call << 6 | info);
      if (lane < 4) put_flag(ws + XF_OFF + gs * 128 + lane * 4, call);
    }
  }
  const unsigned par = (call & 1) * PAR_STRIDE;
  if (threadIdx.x < 64) *(unsigned*)(smem + 32768 + lane * 4) = 0;
  asm volatile("s_waitcnt lgkmcnt(0)\n\ts_barrier" ::: "memory");

  const int nrep = isown ? REPO : ((REPF[0] | REPF[1] << 8 | REPF[2] << 16 | REPF[3] << 24) >> (8 * g)) & 255;
  const int n = __popcll(__ballot(lane * nrep + rep < (isown ? NOWN : NFOR)));
  const unsigned ev = TB.order[type][(lane < n ? lane : 0) * nrep + rep];
  const unsigned foff = XF_OFF + (ev * 8 + wave) * 128;
  const unsigned abase = XS_OFF + par + wave * 16384;
  u32x4 A[16], pf = {0, 0, 0, 0};
#pragma unroll
  for (int j = 0; j < 16; ++j) A[j] = pf;
  bool good = true;
  if (type != 0) {
    good = spin4<8>(rs, ws, foff, lane == 0, call, t0);
    load_block(A, rs, l16, abase + __builtin_amdgcn_readlane(ev, 0) * 131072);
  }
  h16x8 B[16][2];
#pragma unroll
  for (int hf = 0; hf < 2; ++hf) {
    good = spin4<8>(rs, ws, WF_OFF + (cg * 16 + wave * 2 + hf) * 128, lane == 0, call, t0) && good;
    const unsigned so = WS_OFF + par + (cg * 16 + wave * 2) * 16384;
#pragma unroll
    for (int s = hf * 8; s < hf * 8 + 8; ++s)
#pragma unroll
      for (int t = 0; t < 2; ++t)
        B[s][t] = __builtin_bit_cast(h16x8, __builtin_amdgcn_raw_buffer_load_b128(rs, l16 + (s & 1) * 2048 + t * 1024, so + (s >> 1) * 4096, SYS));
  }
  if (type == 0) {
    good = spin4<1>(rs, ws, foff, lane == 0, call, t0) && good;
    load_block(A, rs, l16, abase + __builtin_amdgcn_readlane(ev, 0) * 131072);
  }
  ldbuf<0>(pf, rs, lane >= 1 && lane <= 3 ? foff : OOB, 0);

  char* const ob = isown ? a.out + cg * 128 : ws + OS_OFF + (type - 1) * MiB + g * 128;
  const unsigned wslot = wave * 2048 + (lane & 15) * 128, wrot = (lane >> 4) * 2 + (lane & 15);
  const unsigned rm = wave * 2 + ((lane >> 3) & 1), rc = lane & 7;
  const unsigned rslot = (lane >> 4) * 2048 + rm * 128 + ((rc + rm) & 7) * 16;
  const unsigned ovo = rm * (isown ? 2048 : 512) + rc * 16 + (lane >> 4) * 4;
  lu32* const lcnt = (lu32*)(smem + 32768);
  auto reduce = [&](int i) -> unsigned {
    const unsigned e = __builtin_amdgcn_readlane(ev, i);
    const unsigned o = ovo + (isown ? ((a.rank ^ (e >> 5)) * 512 + (e & 31) * 16) * 2048 : (i * nrep + rep) * 8192);
    const char* const pb = smem + (i & 1) * 16384 + rslot;
    const unsigned goal = 8 * ((i >> 1) + 1);
    __builtin_amdgcn_s_setprio(0);
    while (__hip_atomic_load(lcnt + (i & 1) * 32, __ATOMIC_ACQUIRE, __HIP_MEMORY_SCOPE_WORKGROUP) < goal) { }
    __builtin_amdgcn_s_setprio(1);
    f32x4 v = *(const f32x4*)pb + *(const f32x4*)(pb + 8192);
    const auto a0 = __builtin_amdgcn_permlane32_swap(__float_as_uint(v[0]), __float_as_uint(v[2]), false, false);
    const auto a1 = __builtin_amdgcn_permlane32_swap(__float_as_uint(v[1]), __float_as_uint(v[3]), false, false);
    const float b0 = __uint_as_float(a0[0]) + __uint_as_float(a0[1]), b1 = __uint_as_float(a1[0]) + __uint_as_float(a1[1]);
    const auto c0 = __builtin_amdgcn_permlane16_swap(__float_as_uint(b0), __float_as_uint(b1), false, false);
    const float t = __uint_as_float(c0[0]) + __uint_as_float(c0[1]);
    asm volatile("global_store_dword %0, %1, %2 sc0 sc1" :: "v"(o), "v"(t), "s"(ob) : "memory");
    return isown ? 0 : i + 1;
  };
  auto report = [&](unsigned f0, unsigned f1, unsigned f2) {
    const unsigned f = lane == 0 ? f0 : lane == 1 ? f1 : f2;
    if (lane < 3 && f) {
      if ((__hip_atomic_fetch_add(lcnt + 8 + (f & 3), 1u, __ATOMIC_RELAXED, __HIP_MEMORY_SCOPE_WORKGROUP) & 7) == 7) {
        const unsigned k = (f - 1) * nrep + rep;
        if (k >= 2 * NP) put_flag(ws + SG_OFF + ((type - 1) * 8 + k - 2 * NP) * 128 + g * 4, call);
        else __hip_atomic_fetch_add((gu32*)(ws + RF_OFF + ((call & 1) * 8 + 3 + type) * 1024 + (JO + (k >> 1)) * 4), 1u,
                                    __ATOMIC_RELAXED, __HIP_MEMORY_SCOPE_SYSTEM);
      }
    }
  };
  unsigned long long mask = 0;
  unsigned pend = 0, fresh = 0;
  bool owed = false;
  __builtin_amdgcn_s_waitcnt(0x0F70);
  for (int i = 0; i < n; ++i) {
    const bool more = i + 1 < n;
    const unsigned nbase = abase + __builtin_amdgcn_readlane(ev, more ? i + 1 : i) * 131072;
    asm volatile("s_waitcnt vmcnt(17)" : "+v"(pf) :: "memory");
    mask |= __ballot(all_ge(pf, call));
    const unsigned hv = (unsigned)(mask >> (i + 1)) & (more ? 1u : 0u);
    bool have = hv != 0;
    const unsigned pfo = (lane > i && lane <= i + 3 && !((mask >> lane) & 1)) ? foff : OOB;
    asm volatile("buffer_load_dwordx4 %0, %1, %2, 0 offen sc0 sc1\n\ts_nop 3" : "+v"(pf) : "v"(pfo), "s"(rs) : "memory");
    f32x4 acc0 = {0, 0, 0, 0}, acc1 = {0, 0, 0, 0};
#define MM(j) acc0 = __builtin_amdgcn_mfma_f32_16x16x32_f16(B[j][0], __builtin_bit_cast(h16x8, A[j]), acc0, 0, 0, 0); \
              acc1 = __builtin_amdgcn_mfma_f32_16x16x32_f16(B[j][1], __builtin_bit_cast(h16x8, A[j]), acc1, 0, 0, 0);
#define MS(j) asm volatile("s_cmp_lg_u32 %4, 0\n\ts_cbranch_scc1 1f\n\ts_waitcnt vmcnt(%5)\n1:\n\ts_waitcnt vmcnt(14)"       \
                           : "+v"(A[j]), "+v"(A[j + 1]), "+v"(A[j + 2]), "+v"(A[j + 3]) : "s"(hv), "i"(13 - (j)) : "memory", "scc"); \
    MM(j) MM(j + 1) MM(j + 2) MM(j + 3)                                                                          \
    __builtin_amdgcn_sched_barrier(0);                                                                           \
    asm volatile("s_cmp_lg_u32 %7, 0\n\ts_cbranch_scc0 1f\n\tbuffer_load_dwordx4 %0, %4, %5, %6 offen sc0 sc1\n\t" \
                 "buffer_load_dwordx4 %1, %4, %5, %6 offen offset:1024 sc0 sc1\n\tbuffer_load_dwordx4 %2, %4, %5, %6 offen offset:2048 sc0 sc1\n\t" \
                 "buffer_load_dwordx4 %3, %4, %5, %6 offen offset:3072 sc0 sc1\n1:"                                \
                 : "+v"(A[j]), "+v"(A[j + 1]), "+v"(A[j + 2]), "+v"(A[j + 3]) : "v"(l16), "s"(rs), "s"(nbase + ((j) >> 2) * 4096), "s"(hv) : "memory", "scc"); \
    __builtin_amdgcn_sched_barrier(0);
#define MH(j) MS(j) MS(j + 4)
    MH(0)
    if (have) { WAITV(17); report(pend, 0, 0); pend = 0; }
    if (owed) fresh = reduce(i - 1); else if (have) null_store(rs, l16);
    if (have) { pend = fresh; fresh = 0; }
    __builtin_amdgcn_sched_barrier(0);
    MH(8)
#undef MH
#undef MM
#undef MS
    if (!good) { acc0 = __builtin_nanf(""); acc1 = acc0; }
    char* const pb = smem + (i & 1) * 16384;
    *(f32x4*)(pb + wslot + (wrot & 7) * 16) = f32x4{acc0[0], acc1[0], acc0[1], acc1[1]};
    *(f32x4*)(pb + wslot + ((wrot + 1) & 7) * 16) = f32x4{acc0[2], acc1[2], acc0[3], acc1[3]};
    asm volatile("s_waitcnt lgkmcnt(0)" ::: "memory");
    if (lane == 0) __hip_atomic_fetch_add(lcnt + (i & 1) * 32, 1u, __ATOMIC_RELAXED, __HIP_MEMORY_SCOPE_WORKGROUP);
    owed = have;
    if (!have) {
      if (more) {
        asm volatile("s_waitcnt vmcnt(0)" : "+v"(pf) :: "memory");
        report(pend, fresh, 0);
        pend = fresh = 0;
        mask |= __ballot(all_ge(pf, call));
        if ((mask >> (i + 1)) & 1) { have = true; load_block(A, rs, l16, nbase); }
      }
      const unsigned f = reduce(i);
      if (have) {
        fresh = f;
      } else {
        if (f | pend | fresh) { WAITV(0); report(pend, fresh, f); pend = fresh = 0; }
        if (more) {
          good = spin4<1>(rs, ws, foff, lane == i + 1, call, t0) && good;
          load_block(A, rs, l16, nbase);
          null_store(rs, l16);
        }
      }
    }
  }
  WAITV(0);
}

}

extern "C" void kernel_launch(int rank, int nranks, ncclComm_t comm, hipStream_t stream,
                              void* const* d_in, const int* in_sizes, int n_in,
                              void* d_out, int out_size, const int* in_dtypes, int out_dtype,
                              void* const* d_out_all, void* const* d_ws_all, size_t ws_bytes) {
  if (nranks != 8 || (unsigned)rank >= 8u || n_in < 2 || ws_bytes < WS_USED || out_size != 4096 * 512 ||
      in_sizes[0] != 512 * 4096 || in_sizes[1] != 4096 * 512) {
    fprintf(stderr, "unexpected problem shape\n");
    return;
  }
  constexpr int LDS_BYTES = 96 * 1024;
  HIP_CHECK(hipFuncSetAttribute((const void*)ag_gemm, hipFuncAttributeMaxDynamicSharedMemorySize, LDS_BYTES));
  Args a;
  a.x = (const float*)d_in[0];
  a.w = (const float*)d_in[1];
  for (int d = 0; d < 8; ++d) a.ws[d] = (char*)d_ws_all[rank ^ d];
  a.out = (char*)d_out;
  a.rank = rank;
  a.links = 0;
  int dev = -1, ndev = 0;
  if (hipGetDevice(&dev) == hipSuccess && hipGetDeviceCount(&ndev) == hipSuccess && dev == rank && ndev == 8) {
    const unsigned char upper[8] = {0xB2, 0x8E, 0xE8, 0xD4, 0xD4, 0xE8, 0x8E, 0xB2};
    unsigned lo = 0, hi = 4;
    for (unsigned d = 1; d < 8; ++d) a.links |= d << (4 * ((upper[rank] >> d & 1) ? hi++ : lo++));
  }
  ag_gemm<<<256, 512, LDS_BYTES, stream>>>(a);
  HIP_CHECK(hipGetLastError());
}
```
